```python
import numpy as np

kernel_name = "gpu_a2a_gemm_m4096_k4096_n4096_f32_gelu"


ABSMAX_RTOL = 0.001

COMPUTE_DTYPE = "float32"
EPILOGUE = "gelu"
IN_QUANT = None
OUT_QUANT = None
IN_DTYPES = {"x": "float32", "w_mat": "float32"}
WORLD_SIZE = 8

def setup_inputs():

    x = np.random.randn(4096, 4096).astype(np.float32)
    w_mat = (np.random.randn(4096, 4096) / np.sqrt(4096)).astype(np.float32)
    return {"x": x, "w_mat": w_mat}

def _epilogue(y):
    c = 0.7978845608028654
    return (0.5 * y * (1.0 + np.tanh(c * (y + 0.044715 * y ** 3)))).astype(np.float32)

def reference(x, w_mat):
    y = (x @ w_mat).astype(np.float32)
    return _epilogue(y)
```

```cpp
#include <hip/hip_runtime.h>
#include <rccl/rccl.h>
#include <cstdio>
#include <cstdint>

#define HIP_CHECK(expr) do { hipError_t _e = (expr); if (_e != hipSuccess) { \
    fprintf(stderr, "HIP error %s at %s:%d\n", hipGetErrorString(_e), __FILE__, __LINE__); \
    return; } } while (0)

namespace {

typedef float f4 __attribute__((ext_vector_type(4)));
typedef float f16v __attribute__((ext_vector_type(16)));
typedef _Float16 h8 __attribute__((ext_vector_type(8)));
typedef _Float16 h2 __attribute__((ext_vector_type(2)));
typedef unsigned u4 __attribute__((ext_vector_type(4)));
typedef unsigned u2 __attribute__((ext_vector_type(2)));

constexpr int NR = 8;
constexpr int NSTEP = 16;
constexpr int NWG = 256;
constexpr int NTHREADS = 640;

constexpr size_t SLOT_BYTES = 4u << 20;
constexpr size_t WS_STAGE = 0;
constexpr size_t WS_XBUF = 8u << 20;
constexpr size_t WS_CTL = 40u << 20;
constexpr size_t CTL_CTR = WS_CTL;
constexpr size_t CTL_TMO = WS_CTL + 16384;
constexpr size_t CTL_RFLAG = WS_CTL + 32768;
constexpr size_t CTL_XFLAG = WS_CTL + 65536;
constexpr size_t CTL_XCC = WS_CTL + 131072;
constexpr size_t WS_NEED = WS_CTL + 262144;

constexpr unsigned long long DEADLINE_TICKS = 100000000ull;

struct Params {
    const float* x;
    const float* w;
    float* out;
    char* ws[NR];
    int rank;
};

constexpr int LDS_B = 0;
constexpr int LDS_CTL = 32768;
constexpr int LDS_BYTES = 32768 + 64;

__device__ __forceinline__ unsigned pk(float a, float b) {
    h2 h; h.x = (_Float16)a; h.y = (_Float16)b;
    return __builtin_bit_cast(unsigned, h);
}

#define GLOAD4(dst, voff, sbase, imm) \
    asm volatile("global_load_dwordx4 %0, %1, %2 offset:" #imm : "=v"(dst) : "v"(voff), "s"(sbase) : "memory")

__device__ __forceinline__ unsigned lds_addr(const char* p) {
    return (unsigned)(uintptr_t)(const __attribute__((address_space(3))) char*)p;
}
__device__ __forceinline__ unsigned lds_ld(const char* smem, int word) {
    unsigned v;
    const unsigned a = lds_addr(smem) + LDS_CTL + 4 * word;
    asm volatile("ds_read_b32 %0, %1\n\ts_waitcnt lgkmcnt(0)" : "=v"(v) : "v"(a) : "memory");
    return v;
}
__device__ __forceinline__ void lds_st(char* smem, int word, unsigned v) {
    const unsigned a = lds_addr(smem) + LDS_CTL + 4 * word;
    asm volatile("ds_write_b32 %0, %1" :: "v"(a), "v"(v) : "memory");
}

__device__ __forceinline__ void sw_barrier(char* smem, unsigned target, int lane) {
    asm volatile("s_waitcnt lgkmcnt(0)" ::: "memory");
    if (lane == 0) {
        const unsigned a = lds_addr(smem) + LDS_CTL + 4, one = 1u;
        asm volatile("ds_add_u32 %0, %1" :: "v"(a), "v"(one) : "memory");
    }
    while ((unsigned)__builtin_amdgcn_readfirstlane(lds_ld(smem, 1)) < target) {}
}

__device__ __forceinline__ void step_chunk(int j, int ks, int rank, int& p, int& c) {
    const int q = ks + 4 * j;
    if (q < 8) { p = rank; c = q; }
    else { const int idx = q - 8; c = idx / 7; p = (rank + 1 + (idx - 7 * c)) & 7; }
}

__device__ __forceinline__ float gelu(float y) {
    const float u = 0.7978845608028654f * (y + 0.044715f * y * y * y);
    const float e = __builtin_amdgcn_exp2f(-2.8853900817779268f * u);
    return y * __builtin_amdgcn_rcpf(1.0f + e);
}

__device__ __forceinline__ void w_to_lds(const f4 (&wv)[4], char* bbuf, unsigned boff) {
    unsigned p0a = pk(wv[0].x, wv[1].x), p0b = pk(wv[2].x, wv[3].x);
    unsigned p1a = pk(wv[0].y, wv[1].y), p1b = pk(wv[2].y, wv[3].y);
    unsigned p2a = pk(wv[0].z, wv[1].z), p2b = pk(wv[2].z, wv[3].z);
    unsigned p3a = pk(wv[0].w, wv[1].w), p3b = pk(wv[2].w, wv[3].w);
    auto s0 = __builtin_amdgcn_permlane32_swap(p0a, p2a, false, false);
    auto s1 = __builtin_amdgcn_permlane32_swap(p0b, p2b, false, false);
    auto s2 = __builtin_amdgcn_permlane32_swap(p1a, p3a, false, false);
    auto s3 = __builtin_amdgcn_permlane32_swap(p1b, p3b, false, false);
    u4 fa = {s0[0], s1[0], s0[1], s1[1]};
    u4 fb = {s2[0], s3[0], s2[1], s3[1]};
    *(u4*)(bbuf + boff) = fa;
    *(u4*)(bbuf + boff + 512) = fb;
}

__device__ __forceinline__ u4 x_to_frag(const f4& a, const f4& b) {
    u4 r = {pk(a.x, a.y), pk(a.z, a.w), pk(b.x, b.y), pk(b.z, b.w)};
    return r;
}

template <int S> struct StepTag { static constexpr int value = S; };

__global__ __launch_bounds__(NTHREADS) void a2a_gemm_gelu(Params prm) {
    __shared__ __attribute__((aligned(16))) char smem[LDS_BYTES];
    const int lane = threadIdx.x & 63;
    const int w = __builtin_amdgcn_readfirstlane(threadIdx.x >> 6);
    const int b = blockIdx.x;
    const int xcd = b & 7, jb = b >> 3;
    const int ks = jb & 3, mt = (jb >> 2) & 1, nt = xcd * 4 + (jb >> 3);
    const int rank = prm.rank;
    char* const myws = prm.ws[rank & 7];

    if (w < 8) {
        const int r = lane & 31, hh = lane >> 5;
        unsigned voffW[4];
#pragma unroll
        for (int jj = 0; jj < 4; ++jj) voffW[jj] = (unsigned)(((4 * (2 * w + hh) + jj) * 4096 + nt * 128 + 4 * r) * 4);
        const unsigned voffA = (unsigned)((mt * 8 + w) * 4096 + lane * 16);
        const unsigned voffX = (unsigned)((mt * 256 + 32 * w + r) * 2048 + hh * 32);
        const unsigned boffW = (unsigned)((w * 4 + 2 * hh) * 512 + r * 16);
        const unsigned boffR = (unsigned)(hh * 2048 + r * 16);

        f4 Wr[2][4];
        u4 Ar[2][4];
        f4 Xr[2][8];
        {
            const char* xb0 = (const char*)prm.x + (size_t)rank * (512 * 2048) + ks * 256;
            const char* xb1 = xb0 + 1024;
            const char* wb0 = (const char*)prm.w + (size_t)(rank * 512 + ks * 64) * 16384;
            const char* wb1 = wb0 + (size_t)256 * 16384;
            GLOAD4(Xr[0][0], voffX, xb0, 0);   GLOAD4(Xr[0][1], voffX, xb0, 16);
            GLOAD4(Xr[0][2], voffX, xb0, 64);  GLOAD4(Xr[0][3], voffX, xb0, 80);
            GLOAD4(Xr[0][4], voffX, xb0, 128); GLOAD4(Xr[0][5], voffX, xb0, 144);
            GLOAD4(Xr[0][6], voffX, xb0, 192); GLOAD4(Xr[0][7], voffX, xb0, 208);
#pragma unroll
            for (int jj = 0; jj < 4; ++jj) GLOAD4(Wr[0][jj], voffW[jj], wb0, 0);
            GLOAD4(Xr[1][0], voffX, xb1, 0);   GLOAD4(Xr[1][1], voffX, xb1, 16);
            GLOAD4(Xr[1][2], voffX, xb1, 64);  GLOAD4(Xr[1][3], voffX, xb1, 80);
            GLOAD4(Xr[1][4], voffX, xb1, 128); GLOAD4(Xr[1][5], voffX, xb1, 144);
            GLOAD4(Xr[1][6], voffX, xb1, 192); GLOAD4(Xr[1][7], voffX, xb1, 208);
#pragma unroll
            for (int jj = 0; jj < 4; ++jj) GLOAD4(Wr[1][jj], voffW[jj], wb1, 0);
        }
        __builtin_amdgcn_s_barrier();
        asm volatile("" ::: "memory");
        const unsigned call = __builtin_amdgcn_readfirstlane(lds_ld(smem, 0));
        const char* const stage = myws + WS_STAGE + (size_t)(call & 1u) * SLOT_BYTES;

        asm volatile("s_waitcnt vmcnt(12)"
                     : "+v"(Xr[0][0]), "+v"(Xr[0][1]), "+v"(Xr[0][2]), "+v"(Xr[0][3]), "+v"(Xr[0][4]), "+v"(Xr[0][5]),
                       "+v"(Xr[0][6]), "+v"(Xr[0][7]), "+v"(Wr[0][0]), "+v"(Wr[0][1]), "+v"(Wr[0][2]), "+v"(Wr[0][3])
                     :: "memory");
#pragma unroll
        for (int kb = 0; kb < 4; ++kb) Ar[0][kb] = x_to_frag(Xr[0][2 * kb], Xr[0][2 * kb + 1]);
        w_to_lds(Wr[0], smem + LDS_B, boffW);
        unsigned bar = 8;
        sw_barrier(smem, bar, lane);

        f16v acc[4];
#pragma unroll
        for (int i = 0; i < 4; ++i)
#pragma unroll
            for (int e = 0; e < 16; ++e) acc[i][e] = 0.f;

        auto step = [&](auto tag) {
            constexpr int s = decltype(tag)::value;
            constexpr int cur = s & 1, nxt = cur ^ 1;
            int p2 = 0, c2 = 0;
            if constexpr (s + 2 < NSTEP) {
                step_chunk(s + 2, ks, rank, p2, c2);
                const char* wb = (const char*)prm.w + (size_t)(p2 * 512 + c2 * 64) * 16384;
#pragma unroll
                for (int jj = 0; jj < 4; ++jj) GLOAD4(Wr[cur][jj], voffW[jj], wb, 0);
            }
            if constexpr (s >= 2) {
                constexpr int n = (s + 2 < NSTEP) ? 12 : ((s + 1 < NSTEP) ? 8 : 0);
                asm volatile("s_waitcnt vmcnt(%4)"
                             : "+v"(Ar[cur][0]), "+v"(Ar[cur][1]), "+v"(Ar[cur][2]), "+v"(Ar[cur][3]) : "i"(n) : "memory");
            }
            const char* bb = smem + LDS_B + cur * 16384 + boffR;
#pragma unroll
            for (int kb = 0; kb < 4; ++kb) {
                const h8 a = __builtin_bit_cast(h8, Ar[cur][kb]);
#pragma unroll
                for (int i = 0; i < 4; ++i) {
                    const h8 bf = *(const h8*)(bb + kb * 4096 + i * 512);
                    acc[i] = __builtin_amdgcn_mfma_f32_32x32x16_f16(a, bf, acc[i], 0, 0, 0);
                }
            }
            if constexpr (s + 2 < NSTEP) {
                while ((unsigned)__builtin_amdgcn_readfirstlane(lds_ld(smem, 2)) < (unsigned)(s + 3)) __builtin_amdgcn_s_sleep(1);
                asm volatile("" ::: "memory");
                const char* ab = stage + (size_t)p2 * (512u << 10) + (size_t)c2 * 65536;
                unsigned va = voffA;
                GLOAD4(Ar[cur][0], va, ab, 0);    GLOAD4(Ar[cur][1], va, ab, 1024);
                GLOAD4(Ar[cur][2], va, ab, 2048); GLOAD4(Ar[cur][3], va, ab, 3072);
            }
            if constexpr (s + 1 < NSTEP) {
                if constexpr (s == 0) {
                    asm volatile("s_waitcnt vmcnt(8)"
                                 : "+v"(Xr[1][0]), "+v"(Xr[1][1]), "+v"(Xr[1][2]), "+v"(Xr[1][3]), "+v"(Xr[1][4]),
                                   "+v"(Xr[1][5]), "+v"(Xr[1][6]), "+v"(Xr[1][7]), "+v"(Wr[1][0]), "+v"(Wr[1][1]),
                                   "+v"(Wr[1][2]), "+v"(Wr[1][3])
                                 :: "memory");
#pragma unroll
                    for (int kb = 0; kb < 4; ++kb) Ar[1][kb] = x_to_frag(Xr[1][2 * kb], Xr[1][2 * kb + 1]);
                } else {
                    constexpr int n = (s + 2 < NSTEP) ? 12 : 4;
                    asm volatile("s_waitcnt vmcnt(%4)"
                                 : "+v"(Wr[nxt][0]), "+v"(Wr[nxt][1]), "+v"(Wr[nxt][2]), "+v"(Wr[nxt][3]) : "i"(n) : "memory");
                }
                w_to_lds(Wr[nxt], smem + LDS_B + nxt * 16384, boffW);
                bar += 8;
                sw_barrier(smem, bar, lane);
            }
        };
        step(StepTag<0>{});  step(StepTag<1>{});  step(StepTag<2>{});  step(StepTag<3>{});
        step(StepTag<4>{});  step(StepTag<5>{});  step(StepTag<6>{});  step(StepTag<7>{});
        step(StepTag<8>{});  step(StepTag<9>{});  step(StepTag<10>{}); step(StepTag<11>{});
        step(StepTag<12>{}); step(StepTag<13>{}); step(StepTag<14>{}); step(StepTag<15>{});

        unsigned xst;
        while ((xst = __builtin_amdgcn_readfirstlane(lds_ld(smem, 4))) == 0u) __builtin_amdgcn_s_sleep(1);
        bool bad = __builtin_amdgcn_readfirstlane(lds_ld(smem, 3)) != 0u;
        const float qnan = __builtin_nanf("");
        const int gid = xcd * 8 + (jb >> 2);
        char* const xg = myws + WS_XBUF + (size_t)gid * (16 * 32768) + w * 4096 + lane * 16;
        auto xrs = __builtin_amdgcn_make_buffer_rsrc((void*)(myws + WS_XBUF + (size_t)gid * (16 * 32768)), 0, 16 * 32768, 0x00020000);
        const unsigned xoff = (unsigned)(w * 4096 + lane * 16);
        (void)xg;
        f4 y[4];
#pragma unroll
        for (int kd = 0; kd < 4; ++kd) {
#pragma unroll
            for (int e = 0; e < 4; ++e) {
                f4 v = {acc[0][4 * kd + e], acc[1][4 * kd + e], acc[2][4 * kd + e], acc[3][4 * kd + e]};
                if (bad) v = f4{qnan, qnan, qnan, qnan};
                if (kd == ks) y[e] = v;
                else {
                    const unsigned o = xoff + (unsigned)((ks * 4 + kd) * 32768 + e * 1024);
                    if (xst == 1u) __builtin_amdgcn_raw_buffer_store_b128(__builtin_bit_cast(u4, v), xrs, o, 0, 0);
                    else           __builtin_amdgcn_raw_buffer_store_b128(__builtin_bit_cast(u4, v), xrs, o, 0, 16);
                }
            }
        }
        asm volatile("s_waitcnt vmcnt(0)" ::: "memory");
        unsigned* const xfl = (unsigned*)(myws + CTL_XFLAG);
        if (lane == 0) __hip_atomic_store(xfl + b * 16 + w, call, __ATOMIC_RELAXED, __HIP_MEMORY_SCOPE_AGENT);
        {
            const int pb = b - (ks << 3) + ((lane & 3) << 3);
            const unsigned* pf = xfl + pb * 16 + w;
            unsigned* tmo = (unsigned*)(myws + CTL_TMO);
            const unsigned long long t0 = __builtin_amdgcn_s_memrealtime();
            for (;;) {
                const unsigned v = __hip_atomic_load(pf, __ATOMIC_RELAXED, __HIP_MEMORY_SCOPE_AGENT);
                if (__all((int)(v - call) >= 0)) break;
                if (__hip_atomic_load(tmo, __ATOMIC_RELAXED, __HIP_MEMORY_SCOPE_AGENT) == call ||
                    __builtin_amdgcn_s_memrealtime() - t0 > DEADLINE_TICKS) {
                    if (lane == 0) __hip_atomic_store(tmo, call, __ATOMIC_RELAXED, __HIP_MEMORY_SCOPE_AGENT);
                    bad = true;
                    break;
                }
                __builtin_amdgcn_s_sleep(1);
            }
            asm volatile("" ::: "memory");
        }
        u4 pv[3][4];
#pragma unroll
        for (int t = 0; t < 3; ++t) {
            const int ksrc = (ks + 1 + t) & 3;
#pragma unroll
            for (int e = 0; e < 4; ++e)
                pv[t][e] = __builtin_amdgcn_raw_buffer_load_b128(xrs, xoff + (unsigned)((ksrc * 4 + ks) * 32768 + e * 1024), 0, 16);
        }
        float* orow = prm.out + (size_t)(mt * 256 + 32 * w + 8 * ks + 4 * hh) * 4096 + nt * 128 + 4 * r;
#pragma unroll
        for (int e = 0; e < 4; ++e) {
            f4 v = y[e] + __builtin_bit_cast(f4, pv[0][e]) + __builtin_bit_cast(f4, pv[1][e]) + __builtin_bit_cast(f4, pv[2][e]);
            v.x = gelu(v.x); v.y = gelu(v.y); v.z = gelu(v.z); v.w = gelu(v.w);
            if (bad) v = f4{qnan, qnan, qnan, qnan};
            *(f4*)(orow + (size_t)e * 4096) = v;
        }
    } else if (w == 8) {
        const int g = b & 31, u = b >> 5;
        const int skip = (u + 1) & 7;
        const int r16 = lane & 15, gq = lane >> 4;
        const unsigned voffS = (unsigned)((16 * g + r16) * 2048 + gq * 32);
        f4 xs[7][4];
#pragma unroll
        for (int i = 0; i < 7; ++i) {
            const int c = i + (i >= skip ? 1 : 0);
            const int dest = (rank + 1 + ((u - c) & 7)) & 7;
            const char* xb = (const char*)prm.x + (size_t)dest * (512 * 2048) + c * 256;
            GLOAD4(xs[i][0], voffS, xb, 0);   GLOAD4(xs[i][1], voffS, xb, 16);
            GLOAD4(xs[i][2], voffS, xb, 128); GLOAD4(xs[i][3], voffS, xb, 144);
        }
        __builtin_amdgcn_s_barrier();
        asm volatile("" ::: "memory");
        const unsigned call = __builtin_amdgcn_readfirstlane(lds_ld(smem, 0));
        const unsigned voffD = (unsigned)((g >> 1) * 4096 + (gq >> 1) * 1024 + ((gq & 1) * 32 + (g & 1) * 16 + r16) * 16);
        const size_t sbase = WS_STAGE + (size_t)(call & 1u) * SLOT_BYTES + (size_t)rank * (512u << 10);
        const unsigned voffF = (unsigned)(g * 4);
        auto send = [&](auto tag) {
            constexpr int i = decltype(tag)::value;
            const int c = i + (i >= skip ? 1 : 0);
            const int dest = (rank + 1 + ((u - c) & 7)) & 7;
            const char* db = prm.ws[dest] + sbase + (size_t)c * 65536;
            const u4 f0 = x_to_frag(xs[i][0], xs[i][1]);
            const u4 f1 = x_to_frag(xs[i][2], xs[i][3]);
            unsigned vd = voffD;
            asm volatile("global_store_dwordx4 %0, %1, %2 sc0 sc1\n\ts_nop 1" :: "v"(vd), "v"(f0), "s"(db) : "memory");
            asm volatile("global_store_dwordx4 %0, %1, %2 offset:2048 sc0 sc1\n\ts_nop 1" :: "v"(vd), "v"(f1), "s"(db) : "memory");
        };
        auto flag = [&](auto tag) {
            constexpr int i = decltype(tag)::value;
            const int c = i + (i >= skip ? 1 : 0);
            const int dest = (rank + 1 + ((u - c) & 7)) & 7;
            const char* fb = prm.ws[dest] + CTL_RFLAG + (size_t)(rank * 8 + c) * 128;
            unsigned vf = voffF, cv = call;
            if (lane == 0)
                asm volatile("global_store_dword %0, %1, %2 sc0 sc1\n\ts_nop 1" :: "v"(vf), "v"(cv), "s"(fb) : "memory");
        };
#define XW(i) "+v"(xs[i][0]), "+v"(xs[i][1]), "+v"(xs[i][2]), "+v"(xs[i][3])
        asm volatile("s_waitcnt vmcnt(20)" : XW(0), XW(1) :: "memory");
        send(StepTag<0>{}); send(StepTag<1>{});
        asm volatile("s_waitcnt vmcnt(4)" : XW(2), XW(3), XW(4), XW(5), XW(6) :: "memory");
        asm volatile("s_waitcnt vmcnt(2)" ::: "memory"); flag(StepTag<0>{}); send(StepTag<2>{});
        asm volatile("s_waitcnt vmcnt(3)" ::: "memory"); flag(StepTag<1>{}); send(StepTag<3>{});
        asm volatile("s_waitcnt vmcnt(3)" ::: "memory"); flag(StepTag<2>{}); send(StepTag<4>{});
        asm volatile("s_waitcnt vmcnt(3)" ::: "memory"); flag(StepTag<3>{}); send(StepTag<5>{});
        asm volatile("s_waitcnt vmcnt(3)" ::: "memory"); flag(StepTag<4>{}); send(StepTag<6>{});
        asm volatile("s_waitcnt vmcnt(3)" ::: "memory"); flag(StepTag<5>{});
        asm volatile("s_waitcnt vmcnt(1)" ::: "memory"); flag(StepTag<6>{});
#undef XW
    } else {
        unsigned* ctr = (unsigned*)(myws + CTL_CTR) + b * 16;
        const unsigned call = __builtin_amdgcn_readfirstlane(__hip_atomic_load(ctr, __ATOMIC_RELAXED, __HIP_MEMORY_SCOPE_AGENT)) + 1u;
        const unsigned myxcc = (unsigned)__builtin_amdgcn_s_getreg((3 << 11) | 20) & 15u;
        unsigned* xcc = (unsigned*)(myws + CTL_XCC);
        if (lane == 0) {
            lds_st(smem, 0, call); lds_st(smem, 1, 0u); lds_st(smem, 2, 2u); lds_st(smem, 3, 0u); lds_st(smem, 4, 0u);
            __hip_atomic_store(ctr, call, __ATOMIC_RELAXED, __HIP_MEMORY_SCOPE_AGENT);
            __hip_atomic_store(xcc + b * 16, (call << 4) | myxcc, __ATOMIC_RELAXED, __HIP_MEMORY_SCOPE_AGENT);
        }
        asm volatile("s_waitcnt lgkmcnt(0)" ::: "memory");
        __builtin_amdgcn_s_barrier();
        asm volatile("" ::: "memory");
        unsigned* tmo = (unsigned*)(myws + CTL_TMO);
        const unsigned long long t0 = __builtin_amdgcn_s_memrealtime();
        bool fail = false;
        {
            const int pb = b - (ks << 3) + ((lane & 3) << 3);
            for (;;) {
                const unsigned v = __hip_atomic_load(xcc + pb * 16, __ATOMIC_RELAXED, __HIP_MEMORY_SCOPE_AGENT);
                if (__all((v >> 4) == (call & 0x0fffffffu))) {
                    const bool same = __all((v & 15u) == myxcc);
                    if (lane == 0) lds_st(smem, 4, same ? 1u : 2u);
                    break;
                }
                if (__hip_atomic_load(tmo, __ATOMIC_RELAXED, __HIP_MEMORY_SCOPE_AGENT) == call ||
                    __builtin_amdgcn_s_memrealtime() - t0 > DEADLINE_TICKS) { fail = true; break; }
                __builtin_amdgcn_s_sleep(1);
            }
        }
        int next = 2;
        const unsigned* rfl = (const unsigned*)(myws + CTL_RFLAG);
        while (!fail && next < NSTEP) {
            const int jj = next + (lane >> 4);
            int p, c;
            step_chunk(jj < NSTEP ? jj : NSTEP - 1, ks, rank, p, c);
            const unsigned v = __hip_atomic_load(rfl + (p * 8 + c) * 32 + mt * 16 + (lane & 15), __ATOMIC_RELAXED, __HIP_MEMORY_SCOPE_SYSTEM);
            const unsigned long long m = __ballot((int)(v - call) >= 0);
            int adv = 0;
            if ((m & 0xffffull) == 0xffffull) { adv = 1;
                if (((m >> 16) & 0xffffull) == 0xffffull) { adv = 2;
                    if (((m >> 32) & 0xffffull) == 0xffffull) { adv = 3;
                        if ((m >> 48) == 0xffffull) adv = 4; } } }
            if (adv) {
                next += adv;
                if (lane == 0) lds_st(smem, 2, (unsigned)next);
            } else {
                if (__hip_atomic_load(tmo, __ATOMIC_RELAXED, __HIP_MEMORY_SCOPE_AGENT) == call ||
                    __builtin_amdgcn_s_memrealtime() - t0 > DEADLINE_TICKS) { fail = true; break; }
                __builtin_amdgcn_s_sleep(1);
            }
        }
        if (fail && lane == 0) {
            __hip_atomic_store(tmo, call, __ATOMIC_RELAXED, __HIP_MEMORY_SCOPE_AGENT);
            lds_st(smem, 3, 1u);
            asm volatile("s_waitcnt lgkmcnt(0)" ::: "memory");
            lds_st(smem, 4, 2u);
            lds_st(smem, 2, 99u);
        }
    }
}

}

extern "C" void kernel_launch(int rank, int nranks, ncclComm_t comm,
                              hipStream_t stream,
                              void* const* d_in, const int* in_sizes, int n_in,
                              void* d_out, int out_size,
                              const int* in_dtypes, int out_dtype,
                              void* const* d_out_all,
                              void* const* d_ws_all, size_t ws_bytes) {
    (void)comm; (void)in_sizes; (void)n_in; (void)out_size; (void)in_dtypes; (void)out_dtype; (void)d_out_all;
    if (nranks != NR || (unsigned)rank >= (unsigned)NR || ws_bytes < WS_NEED) {
        fprintf(stderr, "unexpected configuration\n");
        return;
    }
    Params prm;
    prm.x = (const float*)d_in[0];
    prm.w = (const float*)d_in[1];
    prm.out = (float*)d_out;
    for (int p = 0; p < NR; ++p) prm.ws[p] = (char*)d_ws_all[p];
    prm.rank = rank;
    hipLaunchKernelGGL(a2a_gemm_gelu, dim3(NWG), dim3(NTHREADS), 0, stream, prm);
    HIP_CHECK(hipGetLastError());
}
```

```cpp
#include <hip/hip_runtime.h>
#include <rccl/rccl.h>
#include <cstdio>
#include <cstdint>

#define HIP_CHECK(expr) do { hipError_t _e = (expr); if (_e != hipSuccess) { \
    fprintf(stderr, "HIP error %s at %s:%d\n", hipGetErrorString(_e), __FILE__, __LINE__); \
    return; } } while (0)

namespace {

typedef float f4 __attribute__((ext_vector_type(4)));
typedef float f16v __attribute__((ext_vector_type(16)));
typedef _Float16 h8 __attribute__((ext_vector_type(8)));
typedef _Float16 h2 __attribute__((ext_vector_type(2)));
typedef unsigned u4 __attribute__((ext_vector_type(4)));

constexpr int NR = 8;
constexpr int NSTEP = 16;
constexpr int NWG = 256;
constexpr int NTHREADS = 512;
constexpr int RING = 8;
#define PACE_TICKS 125
#define PACE_START 100
#define W_DELAY 0
#define WSETS 1
#define FIFO_D 2
#define XAUX 0
#define OAUX 0

constexpr size_t SLOT_BYTES = 4u << 20;
constexpr size_t WS_STAGE = 0;
constexpr size_t WS_XBUF = 8u << 20;
constexpr size_t WS_CTL = 40u << 20;
constexpr size_t CTL_CTR = WS_CTL;
constexpr size_t CTL_TMO = WS_CTL + 16384;
constexpr size_t CTL_RFLAG = WS_CTL + 32768;
constexpr size_t CTL_XFLAG = WS_CTL + 65536;
constexpr size_t CTL_XCC = WS_CTL + 131072;
constexpr size_t WS_NEED = WS_CTL + 262144;

constexpr unsigned long long DEADLINE_TICKS = 100000000ull;

constexpr int LDS_B = 0;
constexpr int LDS_CTL = RING * 16384;
constexpr int LDS_BYTES = LDS_CTL + 128 + 512;

#define STAMP(idx) do {} while (0)
#define STAMPC(idx) do {} while (0)
#define STAMP_DUMP(lo, hi) do {} while (0)

struct Params {
    const float* x;
    const float* w;
    float* out;
    char* ws[NR];
    int rank;
};

__device__ __forceinline__ unsigned pk(float a, float b) {
    h2 h; h.x = (_Float16)a; h.y = (_Float16)b;
    return __builtin_bit_cast(unsigned, h);
}

#define GLOAD4(dst, voff, sbase, imm) \
    asm volatile("global_load_dwordx4 %0, %1, %2 offset:" #imm : "+v"(dst) : "v"(voff), "s"(sbase) : "memory")

__device__ __forceinline__ unsigned lds_addr(const char* p) {
    return (unsigned)(uintptr_t)(const __attribute__((address_space(3))) char*)p;
}
__device__ __forceinline__ unsigned lds_ld(const char* smem, int word) {
    unsigned v;
    const unsigned a = lds_addr(smem) + LDS_CTL + 4 * word;
    asm volatile("ds_read_b32 %0, %1\n\ts_waitcnt lgkmcnt(0)" : "=v"(v) : "v"(a) : "memory");
    return (unsigned)__builtin_amdgcn_readfirstlane(v);
}
__device__ __forceinline__ void lds_st(char* smem, int word, unsigned v) {
    const unsigned a = lds_addr(smem) + LDS_CTL + 4 * word;
    asm volatile("ds_write_b32 %0, %1" :: "v"(a), "v"(v) : "memory");
}
__device__ __forceinline__ void lds_inc(char* smem, int word, int lane) {
    asm volatile("s_waitcnt lgkmcnt(0)" ::: "memory");
    if (lane == 0) {
        const unsigned a = lds_addr(smem) + LDS_CTL + 4 * word, one = 1u;
        asm volatile("ds_add_u32 %0, %1" :: "v"(a), "v"(one) : "memory");
    }
}
__device__ __forceinline__ void lds_wait(const char* smem, int word, unsigned target) {
    while (lds_ld(smem, word) < target) __builtin_amdgcn_s_sleep(1);
}

__device__ __forceinline__ void step_chunk(int j, int ks, int rank, int& p, int& c) {
    const int q = ks + 4 * j;
    if (q < 8) { p = rank; c = q; }
    else { const int idx = q - 8; c = idx / 7; p = (rank + 1 + (idx - 7 * c)) & 7; }
}

__device__ __forceinline__ float gelu(float y) {
    const float u = 0.7978845608028654f * (y + 0.044715f * y * y * y);
    const float e = __builtin_amdgcn_exp2f(-2.8853900817779268f * u);
    return y * __builtin_amdgcn_rcpf(1.0f + e);
}

__device__ __forceinline__ u4 x_to_frag(const f4& a, const f4& b) {
    u4 r = {pk(a.x, a.y), pk(a.z, a.w), pk(b.x, b.y), pk(b.z, b.w)};
    return r;
}

template <int S> struct Tag { static constexpr int value = S; };

__global__ __launch_bounds__(NTHREADS) void a2a_gemm_gelu(Params prm) {
    __shared__ __attribute__((aligned(16))) char smem[LDS_BYTES];
    const int lane = threadIdx.x & 63;
    const int w = __builtin_amdgcn_readfirstlane(threadIdx.x >> 6);
    const int b = blockIdx.x;
    const int xcd = b & 7, jb = b >> 3;
    const int ks = jb & 3, mt = (jb >> 2) & 1, nt = xcd * 4 + (jb >> 3);
    const int rank = prm.rank;
    char* const myws = prm.ws[rank & 7];
    const int r = lane & 31, hh = lane >> 5;

    if (w < 4) {
        if (w == 0) STAMP(0);
        const unsigned voffA = (unsigned)((mt * 16 + 4 * w) * 2048 + lane * 16);
        const unsigned voffA1 = voffA + 4096u;
        const unsigned boffR = (unsigned)(lane * 16);
        __builtin_amdgcn_s_barrier();

        f4 acc[4][8];
#pragma unroll
        for (int a = 0; a < 4; ++a)
#pragma unroll
            for (int t = 0; t < 8; ++t) acc[a][t] = f4{0.f, 0.f, 0.f, 0.f};
        u4 Ar[2][8] = {};
        h8 bf[4];
        typedef const volatile __attribute__((address_space(3))) unsigned* lds_cvu;
        const lds_cvu ctl = (lds_cvu)(smem + LDS_CTL);
        lds_wait(smem, 1, 1u);
        const unsigned call = lds_ld(smem, 0);
        const char* const stage = myws + WS_STAGE + (size_t)(call & 1u) * SLOT_BYTES;

        auto aaddr = [&](int j) -> const char* {
            int p, c;
            step_chunk(j, ks, rank, p, c);
            return stage + ((size_t)(p * 8 + c) << 16);
        };
#define SB __builtin_amdgcn_sched_barrier(0)
#define MF(a, i) asm volatile("v_mfma_f32_16x16x32_f16 %0, %1, %2, %0" : "+v"(acc[a][4 * (gi & 1) + i]) \
            : "v"(A[4 * (gi >> 1) + a]), "v"(bf[i])); SB
        int nc = (ks + 4) / 7, nd = (ks + 4) % 7;
        auto step = [&](u4 (&A)[8], auto ltag, int s, const char*& ab) {
            constexpr int left = decltype(ltag)::value;
            const int slot = s & (RING - 1), nslot = (s + 1) & (RING - 1);
            const char* bb = smem + LDS_B + slot * 16384 + boffR;
            const char* bn = smem + LDS_B + nslot * 16384 + boffR;
            unsigned va = voffA, vb = voffA1;
            unsigned c_full = 0, c_rdy = 0;
            const char* abnext = ab;
#pragma unroll
            for (int gi = 0; gi < 4; ++gi) {
                const char* src = gi < 3 ? bb + (gi + 1) * 4096 : bn;
                const bool rd = gi < 3 || left > 0;
                const bool rl = left == 2 && (gi & 1);
                if ((gi & 1) == 0) {
                    asm volatile("s_waitcnt vmcnt(%4)"
                                 : "+v"(A[2 * gi]), "+v"(A[2 * gi + 1]), "+v"(A[2 * gi + 2]), "+v"(A[2 * gi + 3])
                                 : "n"(left == 2 ? 12 : 8 * left + 4 - 2 * gi) : "memory");
                    SB;
                }
                if (gi == 3) {
                    MF(0, 0); MF(1, 0);
                    lds_inc(smem, 16 + slot, lane);
                    if (left > 0) {
                        const unsigned need = (unsigned)(2 * ((s + 1) / RING + 1));
                        if ((unsigned)__builtin_amdgcn_readfirstlane(c_full) < need) lds_wait(smem, 8 + nslot, need);
                    }
                    SB;
                    MF(2, 0); MF(3, 0);
                } else {
                    MF(0, 0); MF(1, 0); MF(2, 0); MF(3, 0);
                }
                if (rd) { bf[0] = *(const h8*)(src); SB; }
                MF(0, 1); MF(1, 1);
                if (gi == 2 && left > 0) { c_full = ctl[8 + nslot]; c_rdy = ctl[1]; SB; }
                if (gi == 3 && left == 2) {
                    abnext = stage + ((size_t)((((rank + 1 + nd) & 7) * 8 + nc)) << 16);
                    nd += 4; if (nd >= 7) { nd -= 7; nc += 1; }
                    SB;
                }
                MF(2, 1); MF(3, 1);
                if (rd) { bf[1] = *(const h8*)(src + 1024); SB; }
                MF(0, 2); MF(1, 2); MF(2, 2); MF(3, 2);
                if (rd) { bf[2] = *(const h8*)(src + 2048); SB; }
                MF(0, 3);
                if (rl) { if (gi == 1) GLOAD4(A[0], va, ab, 0); else GLOAD4(A[4], va, ab, 1024); SB; }
                MF(1, 3);
                if (rl) { if (gi == 1) GLOAD4(A[1], va, ab, 2048); else GLOAD4(A[5], va, ab, 3072); SB; }
                MF(2, 3);
                if (rl) { if (gi == 1) GLOAD4(A[2], vb, ab, 0); else GLOAD4(A[6], vb, ab, 1024); SB; }
                MF(3, 3);
                if (rl) { if (gi == 1) GLOAD4(A[3], vb, ab, 2048); else GLOAD4(A[7], vb, ab, 3072); SB; }
                if (rd) { bf[3] = *(const h8*)(src + 3072); SB; }
            }
            if (left == 2 && s + 3 < NSTEP) {
                if ((unsigned)__builtin_amdgcn_readfirstlane(c_rdy) < (unsigned)(s + 4)) lds_wait(smem, 1, (unsigned)(s + 4));
            }
            ab = abnext;
        };
#pragma unroll
        for (int t = 0; t < 2; ++t) {
            lds_wait(smem, 1, (unsigned)(t + 1));
            const char* ab0 = aaddr(t);
            GLOAD4(Ar[t][0], voffA, ab0, 0);     GLOAD4(Ar[t][1], voffA, ab0, 2048);
            GLOAD4(Ar[t][2], voffA1, ab0, 0);    GLOAD4(Ar[t][3], voffA1, ab0, 2048);
            GLOAD4(Ar[t][4], voffA, ab0, 1024);  GLOAD4(Ar[t][5], voffA, ab0, 3072);
            GLOAD4(Ar[t][6], voffA1, ab0, 1024); GLOAD4(Ar[t][7], voffA1, ab0, 3072);
        }
        lds_wait(smem, 8, 2u);
#pragma unroll
        for (int i = 0; i < 4; ++i) bf[i] = *(const h8*)(smem + LDS_B + boffR + i * 1024);
        lds_wait(smem, 1, 3u);
        const char* abn = aaddr(2);
        for (int s = 0; s < NSTEP - 2; s += 2) {
            step(Ar[0], Tag<2>{}, s, abn);
            step(Ar[1], Tag<2>{}, s + 1, abn);
        }
        step(Ar[0], Tag<1>{}, NSTEP - 2, abn);
        step(Ar[1], Tag<0>{}, NSTEP - 1, abn);
        asm volatile("s_nop 15\n\ts_nop 7" ::: "memory");
#undef MF
#undef SB

        unsigned xst;
        while ((xst = lds_ld(smem, 3)) == 0u) __builtin_amdgcn_s_sleep(1);
        bool bad = lds_ld(smem, 2) != 0u;
        const float qnan = __builtin_nanf("");
        const int b0 = b - (ks << 3);
        auto xrs = __builtin_amdgcn_make_buffer_rsrc((void*)(myws + WS_XBUF), 0, NWG * 131072, 0x00020000);
        const unsigned xoff = (unsigned)(w * 8192 + lane * 16);
        f4 y[8];
#pragma unroll
        for (int kd = 0; kd < 4; ++kd) {
#pragma unroll
            for (int e = 0; e < 8; ++e) {
                const int G4 = 4 * (e >> 2), g = e & 3;
                f4 v = {acc[kd][G4][g], acc[kd][G4 + 1][g], acc[kd][G4 + 2][g], acc[kd][G4 + 3][g]};
                if (bad) v = f4{qnan, qnan, qnan, qnan};
                if (kd == ks) y[e] = v;
                else {
                    const unsigned o = xoff + (unsigned)(b * 131072 + kd * 32768 + e * 1024);
                    if (xst == 1u) __builtin_amdgcn_raw_buffer_store_b128(__builtin_bit_cast(u4, v), xrs, o, 0, XAUX);
                    else           __builtin_amdgcn_raw_buffer_store_b128(__builtin_bit_cast(u4, v), xrs, o, 0, 16);
                }
            }
        }
        asm volatile("s_waitcnt vmcnt(0)" ::: "memory");
        if (w == 0) STAMP(18);
        unsigned* const xfl = (unsigned*)(myws + CTL_XFLAG);
        if (lane == 0) __hip_atomic_store(xfl + b * 16 + w, call, __ATOMIC_RELAXED, __HIP_MEMORY_SCOPE_AGENT);
        {
            const unsigned* pf = xfl + (b0 + ((lane & 3) << 3)) * 16 + w;
            unsigned* tmo = (unsigned*)(myws + CTL_TMO);
            const unsigned long long t0 = __builtin_amdgcn_s_memrealtime();
            unsigned spins = 0;
            for (;;) {
                const unsigned v = __hip_atomic_load(pf, __ATOMIC_RELAXED, __HIP_MEMORY_SCOPE_AGENT);
                if (__all((int)(v - call) >= 0)) break;
                if (__builtin_amdgcn_s_memrealtime() - t0 > DEADLINE_TICKS ||
                    ((++spins & 63u) == 0u && __hip_atomic_load(tmo, __ATOMIC_RELAXED, __HIP_MEMORY_SCOPE_AGENT) == call)) {
                    if (lane == 0) __hip_atomic_store(tmo, call, __ATOMIC_RELAXED, __HIP_MEMORY_SCOPE_AGENT);
                    bad = true;
                    break;
                }
                __builtin_amdgcn_s_sleep(1);
            }
            asm volatile("" ::: "memory");
        }
        if (w == 0) STAMP(19);
        u4 pv[3][8];
#pragma unroll
        for (int t = 0; t < 3; ++t) {
            const int sb = b0 + (((ks + 1 + t) & 3) << 3);
#pragma unroll
            for (int e = 0; e < 8; ++e)
                pv[t][e] = __builtin_amdgcn_raw_buffer_load_b128(xrs, xoff + (unsigned)(sb * 131072 + ks * 32768 + e * 1024), 0, 16);
        }
        auto ors = __builtin_amdgcn_make_buffer_rsrc((void*)prm.out, 0, 512 * 4096 * 4, 0x00020000);
        const unsigned ooff = (unsigned)(((mt * 256 + 64 * w + 16 * ks + 4 * (lane >> 4)) * 4096 + nt * 128 + 4 * (lane & 15)) * 4);
#pragma unroll
        for (int e = 0; e < 8; ++e) {
            f4 v = y[e] + __builtin_bit_cast(f4, pv[0][e]) + __builtin_bit_cast(f4, pv[1][e]) + __builtin_bit_cast(f4, pv[2][e]);
            v.x = gelu(v.x); v.y = gelu(v.y); v.z = gelu(v.z); v.w = gelu(v.w);
            if (bad) v = f4{qnan, qnan, qnan, qnan};
            __builtin_amdgcn_raw_buffer_store_b128(__builtin_bit_cast(u4, v), ors, ooff + (unsigned)((e & 3) * 16384 + (e >> 2) * 256), 0, OAUX);
        }
    } else if (w < 6) {
        const int pw = w - 4;
        const unsigned voffW0 = (unsigned)(((32 * pw + 8 * hh) * 4096 + nt * 128 + 4 * r) * 4);
        const unsigned boffW = (unsigned)(pw * 8192 + (r >> 4) * 4096 + (hh * 16 + (r & 15)) * 16);
        constexpr int WS = WSETS;
        f4 Wr[WS][8][2] = {};
        u4 F[FIFO_D][2][4] = {};
        static_assert(FIFO_D % WS == 0, "unroll must line up the register sets");
        auto wbase = [&](int j) -> const char* {
            int p, c;
            step_chunk(j, ks, rank, p, c);
            return (const char*)prm.w + (size_t)(p * 512 + c * 64) * 16384;
        };
#define WLOAD(Wv, wb, t) do { const char* _p = (wb) + (size_t)(16 * ((t) >> 2) + 2 * ((t) & 3)) * 16384; const char* _q = _p + 16384; \
            unsigned _v = voffW0; GLOAD4(Wv[t][0], _v, _p, 0); GLOAD4(Wv[t][1], _v, _q, 0); } while (0)
#pragma unroll
        for (int q = 0; q < WS; ++q) {
            const char* wb0 = wbase(q);
#pragma unroll
            for (int t = 0; t < 8; ++t) WLOAD(Wr[q], wb0, t);
        }
        __builtin_amdgcn_s_barrier();
        for (int s0 = 0; s0 < NSTEP + FIFO_D - 1; s0 += FIFO_D) {
#pragma unroll
            for (int su = 0; su < FIFO_D; ++su) {
                const int s = s0 + su;
                if (s < NSTEP) {
                    const char* wn = wbase(s + WS < NSTEP ? s + WS : NSTEP - 1);
                    f4 (&Wv)[8][2] = Wr[su % WS];
#pragma unroll
                    for (int t = 0; t < 8; ++t) {
                        asm volatile("s_waitcnt vmcnt(%2)" : "+v"(Wv[t][0]), "+v"(Wv[t][1]) : "n"(16 * WS - 2) : "memory");
#pragma unroll
                        for (int i = 0; i < 4; ++i) F[su][t >> 2][i][t & 3] = pk(Wv[t][0][i], Wv[t][1][i]);
                        asm volatile("" : "+v"(F[su][t >> 2][0][t & 3]), "+v"(F[su][t >> 2][1][t & 3]),
                                          "+v"(F[su][t >> 2][2][t & 3]), "+v"(F[su][t >> 2][3][t & 3]));
                        __builtin_amdgcn_sched_barrier(0);
                        WLOAD(Wv, wn, t);
                    }
                }
                const int sw = s - (FIFO_D - 1);
                if (sw >= 0 && sw < NSTEP) {
                    const int slot = sw & (RING - 1);
                    if (sw >= RING) lds_wait(smem, 16 + slot, (unsigned)(4 * (sw / RING)));
                    char* bb = smem + LDS_B + slot * 16384 + boffW;
#pragma unroll
                    for (int h2i = 0; h2i < 2; ++h2i)
#pragma unroll
                        for (int i = 0; i < 4; ++i) *(u4*)(bb + h2i * 512 + i * 1024) = F[(su + 1) % FIFO_D][h2i][i];
                    lds_inc(smem, 8 + slot, lane);
                }
            }
        }
#undef WLOAD
        asm volatile("s_waitcnt vmcnt(0)" ::: "memory");
#pragma unroll
        for (int q = 0; q < WS; ++q)
#pragma unroll
            for (int t = 0; t < 8; ++t) asm volatile("" : "+v"(Wr[q][t][0]), "+v"(Wr[q][t][1]));
        if (w == 4) STAMP_DUMP(53, 61);
    } else if (w == 6) {
        const unsigned long long tstart = __builtin_amdgcn_s_memrealtime();
        const int di = b & 7, c = (b >> 3) & 7, m4 = b >> 6;
        const int dest = (rank + di) & 7;
        const int r16 = lane & 15, gq = lane >> 4;
        const unsigned voffS = (unsigned)(r16 * 2048 + gq * 32);
        const char* xb = (const char*)prm.x + (size_t)(dest * 512 + m4 * 128) * 2048 + c * 256;
        f4 xs[8][4] = {};
#pragma unroll
        for (int g = 0; g < 8; ++g) {
            const char* xg = xb + g * 32768;
            GLOAD4(xs[g][0], voffS, xg, 0);   GLOAD4(xs[g][1], voffS, xg, 16);
            GLOAD4(xs[g][2], voffS, xg, 128); GLOAD4(xs[g][3], voffS, xg, 144);
        }
        __builtin_amdgcn_s_barrier();
        const unsigned voffD = (unsigned)(lane * 16);
        char* const dws = prm.ws[dest];
#define XW(i) "+v"(xs[i][0]), "+v"(xs[i][1]), "+v"(xs[i][2]), "+v"(xs[i][3])
        asm volatile("s_waitcnt vmcnt(0)" : XW(0), XW(1), XW(2), XW(3), XW(4), XW(5), XW(6), XW(7) :: "memory");
#undef XW
        u4 fr[8][2];
#pragma unroll
        for (int g = 0; g < 8; ++g) { fr[g][0] = x_to_frag(xs[g][0], xs[g][1]); fr[g][1] = x_to_frag(xs[g][2], xs[g][3]); }
        lds_wait(smem, 0, 1u);
        const unsigned call = lds_ld(smem, 0);
        const char* db = dws + WS_STAGE + (size_t)(call & 1u) * SLOT_BYTES + (size_t)rank * (512u << 10) + (size_t)c * 65536 + m4 * 16384;
        STAMP(23);
        if (di != 0) {
            const unsigned long long due = tstart + (unsigned long long)(PACE_START + c * PACE_TICKS + (m4 * PACE_TICKS) / 4);
            while (__builtin_amdgcn_s_memrealtime() < due) __builtin_amdgcn_s_sleep(1);
        }
        STAMP(24);
#pragma unroll
        for (int g = 0; g < 8; ++g) {
            const char* dg = db + g * 2048;
            asm volatile("global_store_dwordx4 %0, %1, %2 sc0 sc1\n\ts_nop 1" :: "v"(voffD), "v"(fr[g][0]), "s"(dg) : "memory");
            asm volatile("global_store_dwordx4 %0, %1, %2 offset:1024 sc0 sc1\n\ts_nop 1" :: "v"(voffD), "v"(fr[g][1]), "s"(dg) : "memory");
        }
        asm volatile("s_waitcnt vmcnt(0)" ::: "memory");
        STAMP(25);
        if (lane == 0) {
            const char* fb = dws + CTL_RFLAG + (size_t)((rank * 8 + c) * 4 + m4) * 128;
            const unsigned zero = 0u;
            asm volatile("global_store_dword %0, %1, %2 sc0 sc1\n\ts_nop 1" :: "v"(zero), "v"(call), "s"(fb) : "memory");
        }
        STAMP_DUMP(23, 26);
    } else {
        unsigned* ctr = (unsigned*)(myws + CTL_CTR) + b * 16;
        const unsigned cold = __hip_atomic_load(ctr, __ATOMIC_RELAXED, __HIP_MEMORY_SCOPE_AGENT);
        if (lane < 24) {
            const unsigned a = lds_addr(smem) + LDS_CTL + 4 * lane, v = 0u;
            asm volatile("ds_write_b32 %0, %1" :: "v"(a), "v"(v) : "memory");
        }
        asm volatile("s_waitcnt lgkmcnt(0)" ::: "memory");
        __builtin_amdgcn_s_barrier();
        const unsigned call = __builtin_amdgcn_readfirstlane(cold) + 1u;
        const unsigned myxcc = (unsigned)__builtin_amdgcn_s_getreg((3 << 11) | 20) & 15u;
        unsigned* xcc = (unsigned*)(myws + CTL_XCC);
        if (lane == 0) {
            lds_st(smem, 0, call);
            __hip_atomic_store(ctr, call, __ATOMIC_RELAXED, __HIP_MEMORY_SCOPE_AGENT);
            __hip_atomic_store(xcc + b * 16, (call << 4) | myxcc, __ATOMIC_RELAXED, __HIP_MEMORY_SCOPE_AGENT);
        }
        unsigned* tmo = (unsigned*)(myws + CTL_TMO);
        const unsigned long long t0 = __builtin_amdgcn_s_memrealtime();
        bool fail = false, xdone = false;
        STAMP(32);
        int next = 0;
        const char* rfl = myws + CTL_RFLAG + mt * 256;
        const int b0 = b - (ks << 3);
        unsigned spins = 0;
        unsigned pfd = 0;
        while (next < NSTEP || !xdone) {
            int pp, cc;
            { const int jj = next + ((lane & 15) >> 1); step_chunk(jj < NSTEP ? jj : NSTEP - 1, ks, rank, pp, cc); }
            const unsigned* pa = lane < 16 ? (const unsigned*)(rfl + (pp * 8 + cc) * 512 + (lane & 1) * 128)
                                           : xcc + (b0 + ((lane & 3) << 3)) * 16;
            unsigned pvv = 0;
            if (lane < 20) pvv = __hip_atomic_load(pa, __ATOMIC_RELAXED, __HIP_MEMORY_SCOPE_AGENT);
            const unsigned okm = (unsigned)__ballot((int)(pvv - call) >= 0) & 0xffffu;
            int adv = __builtin_ctz(~okm) >> 1;
            if (adv > NSTEP - next) adv = NSTEP - next;
            bool prog = false;
            if (!xdone) {
                const unsigned vm = (unsigned)(__ballot((pvv >> 4) == (call & 0x0fffffffu)) >> 16) & 15u;
                const unsigned sm = (unsigned)(__ballot((pvv & 15u) == myxcc) >> 16) & 15u;
                if (vm == 15u) {
                    if (lane == 0) lds_st(smem, 3, sm == 15u ? 1u : 2u);
                    xdone = true; prog = true;
                }
            }
            if (adv) {
                next += adv; prog = true;
                if (lane == 0) lds_st(smem, 1, (unsigned)next);
            }
            if (!prog) {
                if (__builtin_amdgcn_s_memrealtime() - t0 > DEADLINE_TICKS) { fail = true; break; }
                if ((++spins & 255u) == 0u &&
                    __hip_atomic_load(tmo, __ATOMIC_RELAXED, __HIP_MEMORY_SCOPE_AGENT) == call) { fail = true; break; }
                __builtin_amdgcn_s_sleep(1);
            }
        }
        asm volatile("s_waitcnt vmcnt(0)" : "+v"(pfd) :: "memory");
        STAMP_DUMP(32, 53);
        if (fail && lane == 0) {
            __hip_atomic_store(tmo, call, __ATOMIC_RELAXED, __HIP_MEMORY_SCOPE_AGENT);
            lds_st(smem, 2, 1u);
            asm volatile("s_waitcnt lgkmcnt(0)" ::: "memory");
            lds_st(smem, 3, 2u);
            lds_st(smem, 1, 99u);
        }
    }
}

}

extern "C" void kernel_launch(int rank, int nranks, ncclComm_t comm,
                              hipStream_t stream,
                              void* const* d_in, const int* in_sizes, int n_in,
                              void* d_out, int out_size,
                              const int* in_dtypes, int out_dtype,
                              void* const* d_out_all,
                              void* const* d_ws_all, size_t ws_bytes) {
    (void)comm; (void)in_sizes; (void)n_in; (void)out_size; (void)in_dtypes; (void)out_dtype; (void)d_out_all;
    if (nranks != NR || (unsigned)rank >= (unsigned)NR || ws_bytes < WS_NEED) {
        fprintf(stderr, "unexpected configuration\n");
        return;
    }
    Params prm;
    prm.x = (const float*)d_in[0];
    prm.w = (const float*)d_in[1];
    prm.out = (float*)d_out;
    for (int p = 0; p < NR; ++p) prm.ws[p] = (char*)d_ws_all[p];
    prm.rank = rank;
    hipLaunchKernelGGL(a2a_gemm_gelu, dim3(NWG), dim3(NTHREADS), 0, stream, prm);
    HIP_CHECK(hipGetLastError());
}
```

```cpp
#include <hip/hip_runtime.h>
#include <rccl/rccl.h>
#include <cstdio>
#include <cstdint>

#define HIP_CHECK(expr) do { hipError_t _e = (expr); if (_e != hipSuccess) { \
    fprintf(stderr, "HIP error %s at %s:%d\n", hipGetErrorString(_e), __FILE__, __LINE__); \
    return; } } while (0)

namespace {

typedef float f4 __attribute__((ext_vector_type(4)));
typedef float f16v __attribute__((ext_vector_type(16)));
typedef _Float16 h8 __attribute__((ext_vector_type(8)));
typedef _Float16 h2 __attribute__((ext_vector_type(2)));
typedef unsigned u4 __attribute__((ext_vector_type(4)));

constexpr int NR = 8;
constexpr int NSTEP = 32;
constexpr int NWG = 256;
constexpr int NTHREADS = 640;
constexpr int RING = 8;
#define PACE_TICKS 140
#define PACE_START 100
#define XLEAD 0
#define WSETS 1
#define W_DELAY 100
#define WS_L 1
#define FIFO_L 2
#define FIFO_D 2
#define OAUX 16

constexpr size_t SLOT_BYTES = 4u << 20;
constexpr size_t WS_STAGE = 0;
constexpr size_t WS_XBUF = 8u << 20;
constexpr size_t WS_CTL = 40u << 20;
constexpr size_t CTL_CTR = WS_CTL;
constexpr size_t CTL_TMO = WS_CTL + 16384;
constexpr size_t CTL_RFLAG = WS_CTL + 32768;
constexpr size_t CTL_XFLAG = WS_CTL + 65536;
constexpr size_t WS_NEED = WS_CTL + 262144;

constexpr unsigned long long DEADLINE_TICKS = 100000000ull;

constexpr int LDS_B = 0;
constexpr int LDS_CTL = RING * 16384;
constexpr int LDS_BYTES = LDS_CTL + 128 + 512;

#define STAMP(idx) do {} while (0)
#define STAMPC(idx) do {} while (0)
#define STAMP_DUMP(lo, hi) do {} while (0)

struct Params {
    const float* x;
    const float* w;
    float* out;
    char* ws[NR];
    int rank;
};

__device__ __forceinline__ unsigned pk(float a, float b) {
    h2 h; h.x = (_Float16)a; h.y = (_Float16)b;
    return __builtin_bit_cast(unsigned, h);
}

#define GLOAD4(dst, voff, sbase, imm) \
    asm volatile("global_load_dwordx4 %0, %1, %2 offset:" #imm : "+v"(dst) : "v"(voff), "s"(sbase) : "memory")

__device__ __forceinline__ unsigned lds_addr(const char* p) {
    return (unsigned)(uintptr_t)(const __attribute__((address_space(3))) char*)p;
}
__device__ __forceinline__ unsigned lds_ld(const char* smem, int word) {
    unsigned v;
    const unsigned a = lds_addr(smem) + LDS_CTL + 4 * word;
    asm volatile("ds_read_b32 %0, %1\n\ts_waitcnt lgkmcnt(0)" : "=v"(v) : "v"(a) : "memory");
    return (unsigned)__builtin_amdgcn_readfirstlane(v);
}
__device__ __forceinline__ void lds_st(char* smem, int word, unsigned v) {
    const unsigned a = lds_addr(smem) + LDS_CTL + 4 * word;
    asm volatile("ds_write_b32 %0, %1" :: "v"(a), "v"(v) : "memory");
}
__device__ __forceinline__ void lds_inc(char* smem, int word, int lane) {
    asm volatile("s_waitcnt lgkmcnt(0)" ::: "memory");
    if (lane == 0) {
        const unsigned a = lds_addr(smem) + LDS_CTL + 4 * word, one = 1u;
        asm volatile("ds_add_u32 %0, %1" :: "v"(a), "v"(one) : "memory");
    }
}
__device__ __forceinline__ void lds_wait(const char* smem, int word, unsigned target) {
    while (lds_ld(smem, word) < target) __builtin_amdgcn_s_sleep(1);
}

__device__ __forceinline__ void step_chunk(int j, int kss, int rank, int& p, int& c) {
    const int q = (kss & 1) + 2 * ((j & ~3) | ((j + (kss >> 1)) & 3));
    if (q < 8) { p = rank; c = q; }
    else { const int idx = q - 8; c = idx / 7; p = (rank + 1 + (idx - 7 * c)) & 7; }
}

__device__ __forceinline__ float gelu(float y) {
    const float u = 0.7978845608028654f * (y + 0.044715f * y * y * y);
    const float e = __builtin_amdgcn_exp2f(-2.8853900817779268f * u);
    return y * __builtin_amdgcn_rcpf(1.0f + e);
}

__device__ __forceinline__ u4 x_to_frag(const f4& a, const f4& b) {
    u4 r = {pk(a.x, a.y), pk(a.z, a.w), pk(b.x, b.y), pk(b.z, b.w)};
    return r;
}

__device__ __forceinline__ const char* uni(const char* p) {
    const unsigned long long v = (unsigned long long)p;
    unsigned lo = __builtin_amdgcn_readfirstlane((unsigned)v), hi = __builtin_amdgcn_readfirstlane((unsigned)(v >> 32));
    asm volatile("s_nop 4" : "+s"(lo), "+s"(hi));
    return (const char*)(((unsigned long long)hi << 32) | lo);
}

template <int S> struct Tag { static constexpr int value = S; };

__global__ __launch_bounds__(NTHREADS) void a2a_gemm_gelu(Params prm) {
    __shared__ __attribute__((aligned(16))) char smem[LDS_BYTES];
    const int lane = threadIdx.x & 63;
    const int w = __builtin_amdgcn_readfirstlane(threadIdx.x >> 6);
    const int b = blockIdx.x;
    const int xcd = b & 7, jb = b >> 3;
    const int ks = jb & 1, mt = (jb >> 1) & 3, nt = xcd * 4 + (jb >> 3);
#define ROT ((mt + (jb >> 3)) & 3)
    const int kss = ks + 2 * (ROT);
    const int rank = prm.rank;
    char* const myws = prm.ws[rank & 7];
    const int r = lane & 31, hh = lane >> 5;

    if (w < 4) {
        if (w == 0) STAMP(0);
        const unsigned voffA = (unsigned)((mt * 8 + 2 * w) * 2048 + lane * 16);
        const unsigned boffR = (unsigned)(lane * 16);
        __builtin_amdgcn_s_barrier();

        f4 acc[2][8];
#pragma unroll
        for (int a = 0; a < 2; ++a)
#pragma unroll
            for (int t = 0; t < 8; ++t) acc[a][t] = f4{0.f, 0.f, 0.f, 0.f};
#pragma unroll
        for (int a = 0; a < 2; ++a)
            asm volatile("" : "+v"(acc[a][0]), "+v"(acc[a][1]), "+v"(acc[a][2]), "+v"(acc[a][3]),
                              "+v"(acc[a][4]), "+v"(acc[a][5]), "+v"(acc[a][6]), "+v"(acc[a][7]));
        constexpr int NA = 3;
        u4 Ar[NA][4] = {};
        h8 bf[2][4];
        typedef const volatile __attribute__((address_space(3))) unsigned* lds_cvu;
        const lds_cvu ctl = (lds_cvu)(smem + LDS_CTL);
        lds_wait(smem, 1, 1u);
        const unsigned call = lds_ld(smem, 0);
        const char* const stage = myws + WS_STAGE + (size_t)(call & 1u) * SLOT_BYTES;

        auto aaddr = [&](int j) -> const char* {
            int p, c;
            step_chunk(j, kss, rank, p, c);
            return uni(stage + ((size_t)(p * 8 + c) << 16));
        };
#define SB __builtin_amdgcn_sched_barrier(0)
#define MF(a, i) asm volatile("v_mfma_f32_16x16x32_f16 %0, %1, %2, %0" : "+v"(acc[a][4 * (gi & 1) + i]) \
            : "v"(A[2 * (gi >> 1) + a]), "v"(bf[gi & 1][i])); SB
        auto step = [&](u4 (&A)[4], auto ltag, auto mtag, int s, const char*& ab) {
            constexpr int left = decltype(ltag)::value;
            constexpr bool more = decltype(mtag)::value != 0;
            constexpr bool look = decltype(mtag)::value == 1;
            const int slot = s & (RING - 1), nslot = (s + 1) & (RING - 1);
            const char* bb = smem + LDS_B + slot * 16384 + boffR;
            const char* bn = smem + LDS_B + nslot * 16384 + boffR;
            unsigned va = voffA;
            unsigned c_full = 0, c_rdy = 0;
            const char* abnext = ab;
#pragma unroll
            for (int gi = 0; gi < 4; ++gi) {
                const char* src = gi < 3 ? bb + (gi + 1) * 4096 : bn;
                const bool rd = gi < 3 || more;
                const bool rl = left == NA && (gi & 1);
                if ((gi & 1) == 0) {
                    asm volatile("s_waitcnt vmcnt(%2)" : "+v"(A[gi]), "+v"(A[gi + 1])
                                 : "n"(left == NA ? 4 * NA - 2 : 4 * left + 2 - gi) : "memory");
                    SB;
                }
                if (gi == 3) {
                    MF(0, 0);
                    lds_inc(smem, 16 + slot, lane);
                    if (more) {
                        const unsigned need = (unsigned)(4 * ((s + 1) / RING + 1));
                        if ((unsigned)__builtin_amdgcn_readfirstlane(c_full) < need) lds_wait(smem, 8 + nslot, need);
                    }
                    SB;
                } else {
                    MF(0, 0);
                }
                if (rd) { bf[(gi + 1) & 1][0] = *(const h8*)(src); SB; }
                MF(0, 1);
                if (rd) { bf[(gi + 1) & 1][1] = *(const h8*)(src + 1024); SB; }
                MF(0, 2);
                if (rd) { bf[(gi + 1) & 1][2] = *(const h8*)(src + 2048); SB; }
                MF(0, 3);
                if (rd) { bf[(gi + 1) & 1][3] = *(const h8*)(src + 3072); SB; }
                MF(1, 0);
                if (rl) { if (gi == 1) GLOAD4(A[0], va, ab, 0); else GLOAD4(A[2], va, ab, 1024); SB; }
                if (gi == 2 && more) { c_full = ctl[8 + nslot]; c_rdy = ctl[1]; SB; }
                MF(1, 1);
                if (gi == 3 && left == NA && look) {
                    abnext = aaddr(s + NA + 1 < NSTEP ? s + NA + 1 : NSTEP - 1);
                    SB;
                }
                MF(1, 2);
                MF(1, 3);
                if (rl) { if (gi == 1) GLOAD4(A[1], va, ab, 2048); else GLOAD4(A[3], va, ab, 3072); SB; }
            }
            if (left == NA && look && s + NA + 1 < NSTEP) {
                if ((unsigned)__builtin_amdgcn_readfirstlane(c_rdy) < (unsigned)(s + NA + 2)) lds_wait(smem, 1, (unsigned)(s + NA + 2));
            }
            ab = abnext;
        };
        auto afill = [&](int s0) {
#pragma unroll
            for (int t = 0; t < NA; ++t) {
                lds_wait(smem, 1, (unsigned)(s0 + t + 1));
                const char* ab0 = aaddr(s0 + t);
                unsigned va = voffA;
                u4 (&A)[4] = Ar[t];
                GLOAD4(A[0], va, ab0, 0);    GLOAD4(A[1], va, ab0, 2048);
                GLOAD4(A[2], va, ab0, 1024); GLOAD4(A[3], va, ab0, 3072);
            }
            lds_wait(smem, 1, (unsigned)(s0 + NA + 1));
            return aaddr(s0 + NA);
        };
        const char* abn = afill(0);
        lds_wait(smem, 8, 4u);
#pragma unroll
        for (int i = 0; i < 4; ++i) bf[0][i] = *(const h8*)(smem + LDS_B + boffR + i * 1024);
        step(Ar[0], Tag<3>{}, Tag<2>{}, 0, abn);
        step(Ar[1], Tag<2>{}, Tag<2>{}, 1, abn);
        step(Ar[2], Tag<1>{}, Tag<2>{}, 2, abn);
        step(Ar[0], Tag<0>{}, Tag<2>{}, 3, abn);
        abn = afill(4);
        for (int s = 4; s < 28; s += 3) {
            step(Ar[0], Tag<3>{}, Tag<1>{}, s, abn);
            step(Ar[1], Tag<3>{}, Tag<1>{}, s + 1, abn);
            step(Ar[2], Tag<3>{}, Tag<1>{}, s + 2, abn);
        }
        step(Ar[0], Tag<3>{}, Tag<1>{}, 28, abn);
        step(Ar[1], Tag<2>{}, Tag<1>{}, 29, abn);
        step(Ar[2], Tag<1>{}, Tag<1>{}, 30, abn);
        step(Ar[0], Tag<0>{}, Tag<0>{}, 31, abn);
        asm volatile("s_nop 15\n\ts_nop 7" ::: "memory");
#undef MF
#undef SB

        bool bad = lds_ld(smem, 2) != 0u;
        const float qnan = __builtin_nanf("");
        const int pb = b ^ 8;
        auto xrs = __builtin_amdgcn_make_buffer_rsrc((void*)(myws + WS_XBUF), 0, NWG * 32768, 0x00020000);
        const unsigned xoff = (unsigned)(w * 8192 + lane * 16);
        f4 y[8];
#pragma unroll
        for (int kd = 0; kd < 2; ++kd) {
#pragma unroll
            for (int e = 0; e < 8; ++e) {
                const int G4 = 4 * (e >> 2), g = e & 3;
                f4 v = {acc[kd][G4][g], acc[kd][G4 + 1][g], acc[kd][G4 + 2][g], acc[kd][G4 + 3][g]};
                if (bad) v = f4{qnan, qnan, qnan, qnan};
                if (kd == ks) y[e] = v;
                else __builtin_amdgcn_raw_buffer_store_b128(__builtin_bit_cast(u4, v), xrs, xoff + (unsigned)(b * 32768 + e * 1024), 0, 16);
            }
        }
        unsigned* const xfl = (unsigned*)(myws + CTL_XFLAG);
        asm volatile("s_waitcnt vmcnt(0)" ::: "memory");
        if (lane == 0) __hip_atomic_store(xfl + b * 16 + w, call, __ATOMIC_RELAXED, __HIP_MEMORY_SCOPE_AGENT);
        if (w == 0) STAMP(18);
        {
            const unsigned* pf = xfl + pb * 16 + w;
            unsigned* tmo = (unsigned*)(myws + CTL_TMO);
            const unsigned long long t0 = __builtin_amdgcn_s_memrealtime();
            unsigned spins = 0;
            while (!bad) {
                const unsigned v = __hip_atomic_load(pf, __ATOMIC_RELAXED, __HIP_MEMORY_SCOPE_AGENT);
                if ((int)(v - call) >= 0) break;
                if (__builtin_amdgcn_s_memrealtime() - t0 > DEADLINE_TICKS ||
                    ((++spins & 63u) == 0u && __hip_atomic_load(tmo, __ATOMIC_RELAXED, __HIP_MEMORY_SCOPE_AGENT) == call)) {
                    if (lane == 0) __hip_atomic_store(tmo, call, __ATOMIC_RELAXED, __HIP_MEMORY_SCOPE_AGENT);
                    bad = true;
                }
                __builtin_amdgcn_s_sleep(1);
            }
            asm volatile("" ::: "memory");
        }
        if (w == 0) STAMP(19);
        u4 pv[8];
#pragma unroll
        for (int e = 0; e < 8; ++e)
            pv[e] = __builtin_amdgcn_raw_buffer_load_b128(xrs, xoff + (unsigned)(pb * 32768 + e * 1024), 0, 16);
        auto ors = __builtin_amdgcn_make_buffer_rsrc((void*)prm.out, 0, 512 * 4096 * 4, 0x00020000);
        const unsigned ooff = (unsigned)(((mt * 128 + 32 * w + 16 * ks + 4 * (lane >> 4)) * 4096 + nt * 128 + 4 * (lane & 15)) * 4);
#pragma unroll
        for (int e = 0; e < 8; ++e) {
            f4 v = y[e] + __builtin_bit_cast(f4, pv[e]);
            v.x = gelu(v.x); v.y = gelu(v.y); v.z = gelu(v.z); v.w = gelu(v.w);
            if (bad) v = f4{qnan, qnan, qnan, qnan};
            __builtin_amdgcn_raw_buffer_store_b128(__builtin_bit_cast(u4, v), ors, ooff + (unsigned)((e & 3) * 16384 + (e >> 2) * 256), 0, OAUX);
        }
    } else if (w < 8) {
        const int pw = w - 4;
        unsigned voffW[8];
#pragma unroll
        for (int jj = 0; jj < 8; ++jj) voffW[jj] = (unsigned)(((16 * pw + 8 * hh + jj) * 4096 + nt * 128 + 4 * r) * 4);
        const unsigned boffW = (unsigned)((pw >> 1) * 8192 + (r >> 4) * 4096 + ((2 * (pw & 1) + hh) * 16 + (r & 15)) * 16);
        auto wbase = [&](int j) -> const char* {
            int p, c;
            step_chunk(j, kss, rank, p, c);
            return (const char*)prm.w + ((size_t)(p * 8 + c) << 20);
        };
        auto run = [&](auto dtag, auto wtag, unsigned (&vo)[8]) {
            constexpr int D = decltype(dtag)::value, WS = decltype(wtag)::value;
            static_assert(D % WS == 0, "unroll must line up the register sets");
            f4 Wr[WS][4][2] = {};
            u4 F[D][4] = {};
            __builtin_amdgcn_s_barrier();
            {
                const unsigned long long tw = __builtin_amdgcn_s_memrealtime();
                while (__builtin_amdgcn_s_memrealtime() - tw < (unsigned long long)W_DELAY) __builtin_amdgcn_s_sleep(2);
            }
#pragma unroll
            for (int q = 0; q < WS; ++q) {
                const char* wb0 = wbase(q);
#pragma unroll
                for (int t = 0; t < 4; ++t) { GLOAD4(Wr[q][t][0], vo[2 * t], wb0, 0); GLOAD4(Wr[q][t][1], vo[2 * t + 1], wb0, 0); }
            }
            for (int s0 = 0; s0 < NSTEP + D - 1; s0 += D) {
#pragma unroll
                for (int su = 0; su < D; ++su) {
                    const int s = s0 + su;
                    if (s < NSTEP) {
                        const char* wn = wbase(s + WS < NSTEP ? s + WS : NSTEP - 1);
                        f4 (&Wv)[4][2] = Wr[su % WS];
#pragma unroll
                        for (int t = 0; t < 4; ++t) {
                            asm volatile("s_waitcnt vmcnt(%2)" : "+v"(Wv[t][0]), "+v"(Wv[t][1]) : "n"(8 * WS - 2) : "memory");
#pragma unroll
                            for (int i = 0; i < 4; ++i) F[su][i][t] = pk(Wv[t][0][i], Wv[t][1][i]);
                            asm volatile("" : "+v"(F[su][0][t]), "+v"(F[su][1][t]), "+v"(F[su][2][t]), "+v"(F[su][3][t]));
                            __builtin_amdgcn_sched_barrier(0);
                            GLOAD4(Wv[t][0], vo[2 * t], wn, 0); GLOAD4(Wv[t][1], vo[2 * t + 1], wn, 0);
                        }
                    }
                    const int sw = s - (D - 1);
                    if (sw >= 0 && sw < NSTEP) {
                        const int slot = sw & (RING - 1);
                        if (sw >= RING) lds_wait(smem, 16 + slot, (unsigned)(4 * (sw / RING)));
                        char* bb = smem + LDS_B + slot * 16384 + boffW;
#pragma unroll
                        for (int i = 0; i < 4; ++i) *(u4*)(bb + i * 1024) = F[(su + 1) % D][i];
                        lds_inc(smem, 8 + slot, lane);
                    }
                }
            }
            asm volatile("s_waitcnt vmcnt(0)" ::: "memory");
#pragma unroll
            for (int q = 0; q < WS; ++q)
#pragma unroll
                for (int t = 0; t < 4; ++t) asm volatile("" : "+v"(Wr[q][t][0]), "+v"(Wr[q][t][1]));
        };
        if (pw == mt) run(Tag<FIFO_L>{}, Tag<WS_L>{}, voffW);
        else run(Tag<FIFO_D>{}, Tag<WSETS>{}, voffW);
        if (w == 4) STAMP_DUMP(53, 61);
    } else if (w == 8) {
        const unsigned long long tstart = __builtin_amdgcn_s_memrealtime();
        const int di = b & 7, c = (b >> 3) & 7, m4 = b >> 6;
        const int dest = (rank + di) & 7;
        const int r16 = lane & 15, gq = lane >> 4;
        const unsigned voffS = (unsigned)(r16 * 2048 + gq * 32);
        const char* xb = (const char*)prm.x + (size_t)(dest * 512 + m4 * 128) * 2048 + c * 256;
        f4 xs[8][4] = {};
        const int grp = (7 * c + ((7 - di) & 7)) >> 3;
        const unsigned long long due = tstart + (unsigned long long)(PACE_START + grp * PACE_TICKS + (m4 * PACE_TICKS) / 4);
#pragma unroll
        for (int g = 0; g < 8; ++g) {
            const char* xg = xb + g * 32768;
            GLOAD4(xs[g][0], voffS, xg, 0);   GLOAD4(xs[g][1], voffS, xg, 16);
            GLOAD4(xs[g][2], voffS, xg, 128); GLOAD4(xs[g][3], voffS, xg, 144);
        }
        __builtin_amdgcn_s_barrier();
        const unsigned voffD = (unsigned)(lane * 16);
        char* const dws = prm.ws[dest];
#define XW(i) "+v"(xs[i][0]), "+v"(xs[i][1]), "+v"(xs[i][2]), "+v"(xs[i][3])
        asm volatile("s_waitcnt vmcnt(0)" : XW(0), XW(1), XW(2), XW(3), XW(4), XW(5), XW(6), XW(7) :: "memory");
#undef XW
        u4 fr[8][2];
#pragma unroll
        for (int g = 0; g < 8; ++g) { fr[g][0] = x_to_frag(xs[g][0], xs[g][1]); fr[g][1] = x_to_frag(xs[g][2], xs[g][3]); }
        lds_wait(smem, 0, 1u);
        const unsigned call = lds_ld(smem, 0);
        const char* db = dws + WS_STAGE + (size_t)(call & 1u) * SLOT_BYTES + (size_t)rank * (512u << 10) + (size_t)c * 65536 + m4 * 16384;
        STAMP(23);
        if (di != 0) {
            while (__builtin_amdgcn_s_memrealtime() < due) __builtin_amdgcn_s_sleep(1);
        }
        STAMP(24);
#pragma unroll
        for (int g = 0; g < 8; ++g) {
            const char* dg = db + g * 2048;
            asm volatile("global_store_dwordx4 %0, %1, %2 sc0 sc1\n\ts_nop 1" :: "v"(voffD), "v"(fr[g][0]), "s"(dg) : "memory");
            asm volatile("global_store_dwordx4 %0, %1, %2 offset:1024 sc0 sc1\n\ts_nop 1" :: "v"(voffD), "v"(fr[g][1]), "s"(dg) : "memory");
        }
        asm volatile("s_waitcnt vmcnt(0)" ::: "memory");
        STAMP(25);
        if (lane == 0) {
            const char* fb = dws + CTL_RFLAG + (size_t)((rank * 8 + c) * 4 + m4) * 128;
            const unsigned zero = 0u;
            asm volatile("global_store_dword %0, %1, %2 sc0 sc1\n\ts_nop 1" :: "v"(zero), "v"(call), "s"(fb) : "memory");
        }
        STAMP_DUMP(23, 26);
    } else {
        unsigned* ctr = (unsigned*)(myws + CTL_CTR) + b * 16;
        const unsigned cold = __hip_atomic_load(ctr, __ATOMIC_RELAXED, __HIP_MEMORY_SCOPE_AGENT);
        if (lane < 24) {
            const unsigned a = lds_addr(smem) + LDS_CTL + 4 * lane, v = 0u;
            asm volatile("ds_write_b32 %0, %1" :: "v"(a), "v"(v) : "memory");
        }
        asm volatile("s_waitcnt lgkmcnt(0)" ::: "memory");
        __builtin_amdgcn_s_barrier();
        const unsigned call = __builtin_amdgcn_readfirstlane(cold) + 1u;
        if (lane == 0) {
            lds_st(smem, 0, call);
            __hip_atomic_store(ctr, call, __ATOMIC_RELAXED, __HIP_MEMORY_SCOPE_AGENT);
        }
        unsigned* tmo = (unsigned*)(myws + CTL_TMO);
        const unsigned long long t0 = __builtin_amdgcn_s_memrealtime();
        bool fail = false;
        STAMP(32);
        int next = 0;
        const char* rfl = myws + CTL_RFLAG + mt * 128;
        unsigned spins = 0;
        while (next < NSTEP) {
            int pp, cc;
            { const int jj = next + (lane & 7); step_chunk(jj < NSTEP ? jj : NSTEP - 1, kss, rank, pp, cc); }
            const unsigned* pa = (const unsigned*)(rfl + (pp * 8 + cc) * 512);
            unsigned pvv = 0;
            if (lane < 8) pvv = __hip_atomic_load(pa, __ATOMIC_RELAXED, __HIP_MEMORY_SCOPE_AGENT);
            const unsigned okm = (unsigned)__ballot((int)(pvv - call) >= 0) & 0xffu;
            int adv = __builtin_ctz(~okm);
            if (adv > NSTEP - next) adv = NSTEP - next;
            bool prog = false;
            if (adv) {
                next += adv; prog = true;
                if (lane == 0) lds_st(smem, 1, (unsigned)next);
            }
            if (!prog) {
                if (__builtin_amdgcn_s_memrealtime() - t0 > DEADLINE_TICKS) { fail = true; break; }
                if ((++spins & 255u) == 0u &&
                    __hip_atomic_load(tmo, __ATOMIC_RELAXED, __HIP_MEMORY_SCOPE_AGENT) == call) { fail = true; break; }
                __builtin_amdgcn_s_sleep(1);
            }
        }
        STAMP_DUMP(32, 53);
        if (fail && lane == 0) {
            __hip_atomic_store(tmo, call, __ATOMIC_RELAXED, __HIP_MEMORY_SCOPE_AGENT);
            lds_st(smem, 2, 1u);
            asm volatile("s_waitcnt lgkmcnt(0)" ::: "memory");
            lds_st(smem, 1, 99u);
        }
    }
}

}

extern "C" void kernel_launch(int rank, int nranks, ncclComm_t comm,
                              hipStream_t stream,
                              void* const* d_in, const int* in_sizes, int n_in,
                              void* d_out, int out_size,
                              const int* in_dtypes, int out_dtype,
                              void* const* d_out_all,
                              void* const* d_ws_all, size_t ws_bytes) {
    (void)comm; (void)in_sizes; (void)n_in; (void)out_size; (void)in_dtypes; (void)out_dtype; (void)d_out_all;
    if (nranks != NR || (unsigned)rank >= (unsigned)NR || ws_bytes < WS_NEED) {
        fprintf(stderr, "unexpected configuration\n");
        return;
    }
    Params prm;
    prm.x = (const float*)d_in[0];
    prm.w = (const float*)d_in[1];
    prm.out = (float*)d_out;
    for (int p = 0; p < NR; ++p) prm.ws[p] = (char*)d_ws_all[p];
    prm.rank = rank;
    hipLaunchKernelGGL(a2a_gemm_gelu, dim3(NWG), dim3(NTHREADS), 0, stream, prm);
    HIP_CHECK(hipGetLastError());
}
```

```cpp
#include <hip/hip_runtime.h>
#include <rccl/rccl.h>
#include <cstdio>
#include <cstdint>

#define HIP_CHECK(expr) do { hipError_t _e = (expr); if (_e != hipSuccess) { \
    fprintf(stderr, "HIP error %s at %s:%d\n", hipGetErrorString(_e), __FILE__, __LINE__); \
    return; } } while (0)

namespace {

typedef float f4 __attribute__((ext_vector_type(4)));
typedef _Float16 h8 __attribute__((ext_vector_type(8)));
typedef _Float16 h2 __attribute__((ext_vector_type(2)));
typedef unsigned u4 __attribute__((ext_vector_type(4)));

constexpr int NR = 8;
constexpr int NSTEP = 32;
constexpr int NWG = 256;
constexpr int NTHREADS = 640;
constexpr int RING = 8;
constexpr int PACE_START = 100;
constexpr int PACE_TICKS = 140;
constexpr int W_DELAY = 100;
constexpr int FIFO_D = 2;

constexpr size_t SLOT_BYTES = 4u << 20;
constexpr size_t WS_STAGE = 0;
constexpr size_t WS_XBUF = 8u << 20;
constexpr size_t WS_CTL = 40u << 20;
constexpr size_t CTL_CTR = WS_CTL;
constexpr size_t CTL_TMO = WS_CTL + 16384;
constexpr size_t CTL_RFLAG = WS_CTL + 32768;
constexpr size_t CTL_XFLAG = WS_CTL + 65536;
constexpr size_t WS_NEED = WS_CTL + 262144;

constexpr unsigned long long DEADLINE_TICKS = 100000000ull;

constexpr int LDS_B = 0;
constexpr int LDS_CTL = RING * 16384;
constexpr int LDS_BYTES = LDS_CTL + 128;


struct Params {
    const float* x;
    const float* w;
    float* out;
    char* ws[NR];
    int rank;
};

__device__ __forceinline__ unsigned pk(float a, float b) {
    h2 h; h.x = (_Float16)a; h.y = (_Float16)b;
    return __builtin_bit_cast(unsigned, h);
}

#define GLOAD4(dst, voff, sbase, imm) \
    asm volatile("global_load_dwordx4 %0, %1, %2 offset:" #imm : "+v"(dst) : "v"(voff), "s"(sbase) : "memory")

__device__ __forceinline__ unsigned lds_addr(const char* p) {
    return (unsigned)(uintptr_t)(const __attribute__((address_space(3))) char*)p;
}
__device__ __forceinline__ unsigned lds_ld(const char* smem, int word) {
    unsigned v;
    const unsigned a = lds_addr(smem) + LDS_CTL + 4 * word;
    asm volatile("ds_read_b32 %0, %1\n\ts_waitcnt lgkmcnt(0)" : "=v"(v) : "v"(a) : "memory");
    return (unsigned)__builtin_amdgcn_readfirstlane(v);
}
__device__ __forceinline__ void lds_st(char* smem, int word, unsigned v) {
    const unsigned a = lds_addr(smem) + LDS_CTL + 4 * word;
    asm volatile("ds_write_b32 %0, %1" :: "v"(a), "v"(v) : "memory");
}
__device__ __forceinline__ void lds_inc(char* smem, int word, int lane) {
    asm volatile("s_waitcnt lgkmcnt(0)" ::: "memory");
    if (lane == 0) {
        const unsigned a = lds_addr(smem) + LDS_CTL + 4 * word, one = 1u;
        asm volatile("ds_add_u32 %0, %1" :: "v"(a), "v"(one) : "memory");
    }
}
__device__ __forceinline__ void lds_wait(const char* smem, int word, unsigned target) {
    while (lds_ld(smem, word) < target) __builtin_amdgcn_s_sleep(1);
}

__device__ __forceinline__ void step_chunk(int j, int kss, int rank, int& p, int& c) {
    const int q = (kss & 1) + 2 * ((j & ~3) | ((j + (kss >> 1)) & 3));
    if (q < 8) { p = rank; c = q; }
    else { const int idx = q - 8; c = idx / 7; p = (rank + 1 + (idx - 7 * c)) & 7; }
}

__device__ __forceinline__ float gelu(float y) {
    const float u = 0.7978845608028654f * (y + 0.044715f * y * y * y);
    const float e = __builtin_amdgcn_exp2f(-2.8853900817779268f * u);
    return y * __builtin_amdgcn_rcpf(1.0f + e);
}

__device__ __forceinline__ u4 x_to_frag(const f4& a, const f4& b) {
    u4 r = {pk(a.x, a.y), pk(a.z, a.w), pk(b.x, b.y), pk(b.z, b.w)};
    return r;
}

__device__ __forceinline__ const char* uni(const char* p) {
    const unsigned long long v = (unsigned long long)p;
    unsigned lo = __builtin_amdgcn_readfirstlane((unsigned)v), hi = __builtin_amdgcn_readfirstlane((unsigned)(v >> 32));
    asm volatile("s_nop 4" : "+s"(lo), "+s"(hi));
    return (const char*)(((unsigned long long)hi << 32) | lo);
}

template <int S> struct Tag { static constexpr int value = S; };

__global__ __launch_bounds__(NTHREADS) void a2a_gemm_gelu(Params prm) {
    __shared__ __attribute__((aligned(16))) char smem[LDS_BYTES];
    const int lane = threadIdx.x & 63;
    const int w = __builtin_amdgcn_readfirstlane(threadIdx.x >> 6);
    const int b = blockIdx.x;
    const int xcd = b & 7, jb = b >> 3;
    const int ks = jb & 1, mt = (jb >> 1) & 3, nt = xcd * 4 + (jb >> 3);
    const int kss = ks + 2 * ((mt + (jb >> 3)) & 3);
    const int rank = prm.rank;
    char* const myws = prm.ws[rank & 7];
    const int r = lane & 31, hh = lane >> 5;

    if (w < 4) {
        const unsigned voffA = (unsigned)((mt * 8 + 2 * w) * 2048 + lane * 16);
        const unsigned boffR = (unsigned)(lane * 16);
        __builtin_amdgcn_s_barrier();

        f4 acc[2][8];
#pragma unroll
        for (int a = 0; a < 2; ++a)
#pragma unroll
            for (int t = 0; t < 8; ++t) acc[a][t] = f4{0.f, 0.f, 0.f, 0.f};
#pragma unroll
        for (int a = 0; a < 2; ++a)
            asm volatile("" : "+v"(acc[a][0]), "+v"(acc[a][1]), "+v"(acc[a][2]), "+v"(acc[a][3]),
                              "+v"(acc[a][4]), "+v"(acc[a][5]), "+v"(acc[a][6]), "+v"(acc[a][7]));
        constexpr int NA = 3;
        u4 Ar[NA][4] = {};
        h8 bf[2][4];
        typedef const volatile __attribute__((address_space(3))) unsigned* lds_cvu;
        const lds_cvu ctl = (lds_cvu)(smem + LDS_CTL);
        lds_wait(smem, 1, 1u);
        const unsigned call = lds_ld(smem, 0);
        const char* const stage = myws + WS_STAGE + (size_t)(call & 1u) * SLOT_BYTES;

        auto aaddr = [&](int j) -> const char* {
            int p, c;
            step_chunk(j, kss, rank, p, c);
            return uni(stage + ((size_t)(p * 8 + c) << 16));
        };
#define SB __builtin_amdgcn_sched_barrier(0)
#define MF(a, i) asm volatile("v_mfma_f32_16x16x32_f16 %0, %1, %2, %0" : "+v"(acc[a][4 * (gi & 1) + i]) \
            : "v"(A[2 * (gi >> 1) + a]), "v"(bf[gi & 1][i])); SB
        auto step = [&](u4 (&A)[4], auto ltag, auto mtag, int s, const char*& ab) {
            constexpr int left = decltype(ltag)::value;
            constexpr bool more = decltype(mtag)::value != 0;
            constexpr bool look = decltype(mtag)::value == 1;
            const int slot = s & (RING - 1), nslot = (s + 1) & (RING - 1);
            const char* bb = smem + LDS_B + slot * 16384 + boffR;
            const char* bn = smem + LDS_B + nslot * 16384 + boffR;
            unsigned va = voffA;
            unsigned c_full = 0, c_rdy = 0;
            const char* abnext = ab;
#pragma unroll
            for (int gi = 0; gi < 4; ++gi) {
                const char* src = gi < 3 ? bb + (gi + 1) * 4096 : bn;
                const bool rd = gi < 3 || more;
                const bool rl = left == NA && (gi & 1);
                if ((gi & 1) == 0) {
                    asm volatile("s_waitcnt vmcnt(%2)" : "+v"(A[gi]), "+v"(A[gi + 1])
                                 : "n"(left == NA ? 4 * NA - 2 : 4 * left + 2 - gi) : "memory");
                    SB;
                }
                if (gi == 3) {
                    MF(0, 0);
                    lds_inc(smem, 16 + slot, lane);
                    if (more) {
                        const unsigned need = (unsigned)(4 * ((s + 1) / RING + 1));
                        if ((unsigned)__builtin_amdgcn_readfirstlane(c_full) < need) lds_wait(smem, 8 + nslot, need);
                    }
                    SB;
                } else {
                    MF(0, 0);
                }
                if (rd) { bf[(gi + 1) & 1][0] = *(const h8*)(src); SB; }
                MF(0, 1);
                if (rd) { bf[(gi + 1) & 1][1] = *(const h8*)(src + 1024); SB; }
                MF(0, 2);
                if (rd) { bf[(gi + 1) & 1][2] = *(const h8*)(src + 2048); SB; }
                MF(0, 3);
                if (rd) { bf[(gi + 1) & 1][3] = *(const h8*)(src + 3072); SB; }
                MF(1, 0);
                if (rl) { if (gi == 1) GLOAD4(A[0], va, ab, 0); else GLOAD4(A[2], va, ab, 1024); SB; }
                if (gi == 2 && more) { c_full = ctl[8 + nslot]; c_rdy = ctl[1]; SB; }
                MF(1, 1);
                if (gi == 3 && left == NA && look) {
                    abnext = aaddr(s + NA + 1 < NSTEP ? s + NA + 1 : NSTEP - 1);
                    SB;
                }
                MF(1, 2);
                MF(1, 3);
                if (rl) { if (gi == 1) GLOAD4(A[1], va, ab, 2048); else GLOAD4(A[3], va, ab, 3072); SB; }
            }
            if (left == NA && look && s + NA + 1 < NSTEP) {
                if ((unsigned)__builtin_amdgcn_readfirstlane(c_rdy) < (unsigned)(s + NA + 2)) lds_wait(smem, 1, (unsigned)(s + NA + 2));
            }
            ab = abnext;
        };
        auto afill = [&](int s0) {
#pragma unroll
            for (int t = 0; t < NA; ++t) {
                lds_wait(smem, 1, (unsigned)(s0 + t + 1));
                const char* ab0 = aaddr(s0 + t);
                unsigned va = voffA;
                u4 (&A)[4] = Ar[t];
                GLOAD4(A[0], va, ab0, 0);    GLOAD4(A[1], va, ab0, 2048);
                GLOAD4(A[2], va, ab0, 1024); GLOAD4(A[3], va, ab0, 3072);
            }
            lds_wait(smem, 1, (unsigned)(s0 + NA + 1));
            return aaddr(s0 + NA);
        };
        const char* abn = afill(0);
        lds_wait(smem, 8, 4u);
#pragma unroll
        for (int i = 0; i < 4; ++i) bf[0][i] = *(const h8*)(smem + LDS_B + boffR + i * 1024);
        step(Ar[0], Tag<3>{}, Tag<2>{}, 0, abn);
        step(Ar[1], Tag<2>{}, Tag<2>{}, 1, abn);
        step(Ar[2], Tag<1>{}, Tag<2>{}, 2, abn);
        step(Ar[0], Tag<0>{}, Tag<2>{}, 3, abn);
        abn = afill(4);
        for (int s = 4; s < 28; s += 3) {
            step(Ar[0], Tag<3>{}, Tag<1>{}, s, abn);
            step(Ar[1], Tag<3>{}, Tag<1>{}, s + 1, abn);
            step(Ar[2], Tag<3>{}, Tag<1>{}, s + 2, abn);
        }
        step(Ar[0], Tag<3>{}, Tag<1>{}, 28, abn);
        step(Ar[1], Tag<2>{}, Tag<1>{}, 29, abn);
        step(Ar[2], Tag<1>{}, Tag<1>{}, 30, abn);
        step(Ar[0], Tag<0>{}, Tag<0>{}, 31, abn);
        asm volatile("s_nop 15\n\ts_nop 7" ::: "memory");
#undef MF
#undef SB

        bool bad = lds_ld(smem, 2) != 0u;
        const float qnan = __builtin_nanf("");
        const int pb = b ^ 8;
        auto xrs = __builtin_amdgcn_make_buffer_rsrc((void*)(myws + WS_XBUF), 0, NWG * 32768, 0x00020000);
        const unsigned xoff = (unsigned)(w * 8192 + lane * 16);
        f4 y[8];
#pragma unroll
        for (int kd = 0; kd < 2; ++kd) {
#pragma unroll
            for (int e = 0; e < 8; ++e) {
                const int G4 = 4 * (e >> 2), g = e & 3;
                f4 v = {acc[kd][G4][g], acc[kd][G4 + 1][g], acc[kd][G4 + 2][g], acc[kd][G4 + 3][g]};
                if (bad) v = f4{qnan, qnan, qnan, qnan};
                if (kd == ks) y[e] = v;
                else __builtin_amdgcn_raw_buffer_store_b128(__builtin_bit_cast(u4, v), xrs, xoff + (unsigned)(b * 32768 + e * 1024), 0, 16);
            }
        }
        unsigned* const xfl = (unsigned*)(myws + CTL_XFLAG);
        asm volatile("s_waitcnt vmcnt(0)" ::: "memory");
        if (lane == 0) __hip_atomic_store(xfl + b * 16 + w, call, __ATOMIC_RELAXED, __HIP_MEMORY_SCOPE_AGENT);
        {
            const unsigned* pf = xfl + pb * 16 + w;
            unsigned* tmo = (unsigned*)(myws + CTL_TMO);
            const unsigned long long t0 = __builtin_amdgcn_s_memrealtime();
            unsigned spins = 0;
            while (!bad) {
                const unsigned v = __hip_atomic_load(pf, __ATOMIC_RELAXED, __HIP_MEMORY_SCOPE_AGENT);
                if ((int)(v - call) >= 0) break;
                if (__builtin_amdgcn_s_memrealtime() - t0 > DEADLINE_TICKS ||
                    ((++spins & 63u) == 0u && __hip_atomic_load(tmo, __ATOMIC_RELAXED, __HIP_MEMORY_SCOPE_AGENT) == call)) {
                    if (lane == 0) __hip_atomic_store(tmo, call, __ATOMIC_RELAXED, __HIP_MEMORY_SCOPE_AGENT);
                    bad = true;
                }
                __builtin_amdgcn_s_sleep(1);
            }
            asm volatile("" ::: "memory");
        }
        u4 pv[8];
#pragma unroll
        for (int e = 0; e < 8; ++e)
            pv[e] = __builtin_amdgcn_raw_buffer_load_b128(xrs, xoff + (unsigned)(pb * 32768 + e * 1024), 0, 16);
        auto ors = __builtin_amdgcn_make_buffer_rsrc((void*)prm.out, 0, 512 * 4096 * 4, 0x00020000);
        const unsigned ooff = (unsigned)(((mt * 128 + 32 * w + 16 * ks + 4 * (lane >> 4)) * 4096 + nt * 128 + 4 * (lane & 15)) * 4);
#pragma unroll
        for (int e = 0; e < 8; ++e) {
            f4 v = y[e] + __builtin_bit_cast(f4, pv[e]);
            v.x = gelu(v.x); v.y = gelu(v.y); v.z = gelu(v.z); v.w = gelu(v.w);
            if (bad) v = f4{qnan, qnan, qnan, qnan};
            __builtin_amdgcn_raw_buffer_store_b128(__builtin_bit_cast(u4, v), ors, ooff + (unsigned)((e & 3) * 16384 + (e >> 2) * 256), 0, 16);
        }
    } else if (w < 8) {
        const int pw = w - 4;
        unsigned voffW[8];
#pragma unroll
        for (int jj = 0; jj < 8; ++jj) voffW[jj] = (unsigned)(((16 * pw + 8 * hh + jj) * 4096 + nt * 128 + 4 * r) * 4);
        const unsigned boffW = (unsigned)((pw >> 1) * 8192 + (r >> 4) * 4096 + ((2 * (pw & 1) + hh) * 16 + (r & 15)) * 16);
        auto wbase = [&](int j) -> const char* {
            int p, c;
            step_chunk(j, kss, rank, p, c);
            return (const char*)prm.w + ((size_t)(p * 8 + c) << 20);
        };
        f4 Wv[4][2] = {};
        u4 F[FIFO_D][4] = {};
        __builtin_amdgcn_s_barrier();
        {
            const unsigned long long tw = __builtin_amdgcn_s_memrealtime();
            while (__builtin_amdgcn_s_memrealtime() - tw < (unsigned long long)W_DELAY) __builtin_amdgcn_s_sleep(2);
        }
        {
            const char* wb0 = wbase(0);
#pragma unroll
            for (int t = 0; t < 4; ++t) { GLOAD4(Wv[t][0], voffW[2 * t], wb0, 0); GLOAD4(Wv[t][1], voffW[2 * t + 1], wb0, 0); }
        }
        for (int s0 = 0; s0 < NSTEP + FIFO_D - 1; s0 += FIFO_D) {
#pragma unroll
            for (int su = 0; su < FIFO_D; ++su) {
                const int s = s0 + su;
                if (s < NSTEP) {
                    const char* wn = wbase(s + 1 < NSTEP ? s + 1 : NSTEP - 1);
#pragma unroll
                    for (int t = 0; t < 4; ++t) {
                        asm volatile("s_waitcnt vmcnt(6)" : "+v"(Wv[t][0]), "+v"(Wv[t][1]) :: "memory");
#pragma unroll
                        for (int i = 0; i < 4; ++i) F[su][i][t] = pk(Wv[t][0][i], Wv[t][1][i]);
                        asm volatile("" : "+v"(F[su][0][t]), "+v"(F[su][1][t]), "+v"(F[su][2][t]), "+v"(F[su][3][t]));
                        __builtin_amdgcn_sched_barrier(0);
                        GLOAD4(Wv[t][0], voffW[2 * t], wn, 0); GLOAD4(Wv[t][1], voffW[2 * t + 1], wn, 0);
                    }
                }
                const int sw = s - (FIFO_D - 1);
                if (sw >= 0 && sw < NSTEP) {
                    const int slot = sw & (RING - 1);
                    if (sw >= RING) lds_wait(smem, 16 + slot, (unsigned)(4 * (sw / RING)));
                    char* bb = smem + LDS_B + slot * 16384 + boffW;
#pragma unroll
                    for (int i = 0; i < 4; ++i) *(u4*)(bb + i * 1024) = F[(su + 1) % FIFO_D][i];
                    lds_inc(smem, 8 + slot, lane);
                }
            }
        }
        asm volatile("s_waitcnt vmcnt(0)" ::: "memory");
#pragma unroll
        for (int t = 0; t < 4; ++t) asm volatile("" : "+v"(Wv[t][0]), "+v"(Wv[t][1]));
    } else if (w == 8) {
        const unsigned long long tstart = __builtin_amdgcn_s_memrealtime();
        const int di = b & 7, c = (b >> 3) & 7, m4 = b >> 6;
        const int dest = (rank + di) & 7;
        const int r16 = lane & 15, gq = lane >> 4;
        const unsigned voffS = (unsigned)(r16 * 2048 + gq * 32);
        const char* xb = (const char*)prm.x + (size_t)(dest * 512 + m4 * 128) * 2048 + c * 256;
        f4 xs[8][4] = {};
        const int grp = (7 * c + ((7 - di) & 7)) >> 3;
        const unsigned long long due = tstart + (unsigned long long)(PACE_START + grp * PACE_TICKS + (m4 * PACE_TICKS) / 4);
#pragma unroll
        for (int g = 0; g < 8; ++g) {
            const char* xg = xb + g * 32768;
            GLOAD4(xs[g][0], voffS, xg, 0);   GLOAD4(xs[g][1], voffS, xg, 16);
            GLOAD4(xs[g][2], voffS, xg, 128); GLOAD4(xs[g][3], voffS, xg, 144);
        }
        __builtin_amdgcn_s_barrier();
        const unsigned voffD = (unsigned)(lane * 16);
        char* const dws = prm.ws[dest];
#define XW(i) "+v"(xs[i][0]), "+v"(xs[i][1]), "+v"(xs[i][2]), "+v"(xs[i][3])
        asm volatile("s_waitcnt vmcnt(0)" : XW(0), XW(1), XW(2), XW(3), XW(4), XW(5), XW(6), XW(7) :: "memory");
#undef XW
        u4 fr[8][2];
#pragma unroll
        for (int g = 0; g < 8; ++g) { fr[g][0] = x_to_frag(xs[g][0], xs[g][1]); fr[g][1] = x_to_frag(xs[g][2], xs[g][3]); }
        lds_wait(smem, 0, 1u);
        const unsigned call = lds_ld(smem, 0);
        const char* db = dws + WS_STAGE + (size_t)(call & 1u) * SLOT_BYTES + (size_t)rank * (512u << 10) + (size_t)c * 65536 + m4 * 16384;
        if (di != 0) {
            while (__builtin_amdgcn_s_memrealtime() < due) __builtin_amdgcn_s_sleep(1);
        }
#pragma unroll
        for (int g = 0; g < 8; ++g) {
            const char* dg = db + g * 2048;
            asm volatile("global_store_dwordx4 %0, %1, %2 sc0 sc1\n\ts_nop 1" :: "v"(voffD), "v"(fr[g][0]), "s"(dg) : "memory");
            asm volatile("global_store_dwordx4 %0, %1, %2 offset:1024 sc0 sc1\n\ts_nop 1" :: "v"(voffD), "v"(fr[g][1]), "s"(dg) : "memory");
        }
        asm volatile("s_waitcnt vmcnt(0)" ::: "memory");
        if (lane == 0) {
            const char* fb = dws + CTL_RFLAG + (size_t)((rank * 8 + c) * 4 + m4) * 128;
            const unsigned zero = 0u;
            asm volatile("global_store_dword %0, %1, %2 sc0 sc1\n\ts_nop 1" :: "v"(zero), "v"(call), "s"(fb) : "memory");
        }
    } else {
        unsigned* ctr = (unsigned*)(myws + CTL_CTR) + b * 16;
        const unsigned cold = __hip_atomic_load(ctr, __ATOMIC_RELAXED, __HIP_MEMORY_SCOPE_AGENT);
        if (lane < 24) {
            const unsigned a = lds_addr(smem) + LDS_CTL + 4 * lane, v = 0u;
            asm volatile("ds_write_b32 %0, %1" :: "v"(a), "v"(v) : "memory");
        }
        asm volatile("s_waitcnt lgkmcnt(0)" ::: "memory");
        __builtin_amdgcn_s_barrier();
        const unsigned call = __builtin_amdgcn_readfirstlane(cold) + 1u;
        if (lane == 0) {
            lds_st(smem, 0, call);
            __hip_atomic_store(ctr, call, __ATOMIC_RELAXED, __HIP_MEMORY_SCOPE_AGENT);
        }
        unsigned* tmo = (unsigned*)(myws + CTL_TMO);
        const unsigned long long t0 = __builtin_amdgcn_s_memrealtime();
        bool fail = false;
        int next = 0;
        const char* rfl = myws + CTL_RFLAG + mt * 128;
        unsigned spins = 0;
        while (next < NSTEP) {
            int pp, cc;
            { const int jj = next + (lane & 7); step_chunk(jj < NSTEP ? jj : NSTEP - 1, kss, rank, pp, cc); }
            const unsigned* pa = (const unsigned*)(rfl + (pp * 8 + cc) * 512);
            unsigned pvv = 0;
            if (lane < 8) pvv = __hip_atomic_load(pa, __ATOMIC_RELAXED, __HIP_MEMORY_SCOPE_AGENT);
            const unsigned okm = (unsigned)__ballot((int)(pvv - call) >= 0) & 0xffu;
            int adv = __builtin_ctz(~okm);
            if (adv > NSTEP - next) adv = NSTEP - next;
            bool prog = false;
            if (adv) {
                next += adv; prog = true;
                if (lane == 0) lds_st(smem, 1, (unsigned)next);
            }
            if (!prog) {
                if (__builtin_amdgcn_s_memrealtime() - t0 > DEADLINE_TICKS) { fail = true; break; }
                if ((++spins & 255u) == 0u &&
                    __hip_atomic_load(tmo, __ATOMIC_RELAXED, __HIP_MEMORY_SCOPE_AGENT) == call) { fail = true; break; }
                __builtin_amdgcn_s_sleep(1);
            }
        }
        if (fail && lane == 0) {
            __hip_atomic_store(tmo, call, __ATOMIC_RELAXED, __HIP_MEMORY_SCOPE_AGENT);
            lds_st(smem, 2, 1u);
            asm volatile("s_waitcnt lgkmcnt(0)" ::: "memory");
            lds_st(smem, 1, 99u);
        }
    }
}

}

extern "C" void kernel_launch(int rank, int nranks, ncclComm_t comm,
                              hipStream_t stream,
                              void* const* d_in, const int* in_sizes, int n_in,
                              void* d_out, int out_size,
                              const int* in_dtypes, int out_dtype,
                              void* const* d_out_all,
                              void* const* d_ws_all, size_t ws_bytes) {
    (void)comm; (void)in_sizes; (void)n_in; (void)out_size; (void)in_dtypes; (void)out_dtype; (void)d_out_all;
    if (nranks != NR || (unsigned)rank >= (unsigned)NR || ws_bytes < WS_NEED) {
        fprintf(stderr, "unexpected configuration\n");
        return;
    }
    Params prm;
    prm.x = (const float*)d_in[0];
    prm.w = (const float*)d_in[1];
    prm.out = (float*)d_out;
    for (int p = 0; p < NR; ++p) prm.ws[p] = (char*)d_ws_all[p];
    prm.rank = rank;
    hipLaunchKernelGGL(a2a_gemm_gelu, dim3(NWG), dim3(NTHREADS), 0, stream, prm);
    HIP_CHECK(hipGetLastError());
}
```
